```python
import numpy as np

kernel_name = "gpu_gemm_a2a_m4096_k4096_n16384_w8a8_dqsilu"


ABSMAX_RTOL = 0.001

COMPUTE_DTYPE = "int8"
EPILOGUE = "dequant_silu"
IN_QUANT = "int8"
OUT_QUANT = None
IN_DTYPES = {"x": "int8", "w_mat": "int8",
             "scale_x": "float32", "scale_w": "float32"}
WORLD_SIZE = 8

def setup_inputs():

    x = np.random.randint(-127, 128, size=(4096, 4096)).astype(np.int8)
    w_mat = np.random.randint(-127, 128, size=(4096, 16384)).astype(np.int8)
    scale_x = (0.02 * (1.0 + np.random.rand(1))).astype(np.float32)
    scale_w = (0.02 * (1.0 + np.random.rand(1))).astype(np.float32)
    return {"x": x, "w_mat": w_mat, "scale_x": scale_x, "scale_w": scale_w}

def reference(x, w_mat, scale_x, scale_w):
    acc = x.astype(np.float64) @ w_mat.astype(np.float64)
    y = (acc.astype(np.float32) * (scale_x * scale_w)).astype(np.float32)
    return (y / (1.0 + np.exp(-np.clip(y, -60.0, 60.0)))).astype(np.float32)
```

```cpp
#include <hip/hip_runtime.h>
#include <rccl/rccl.h>
#include <cstdio>

#define HIP_CHECK(expr) do { hipError_t _e = (expr); if (_e != hipSuccess) { \
    fprintf(stderr, "HIP error %s at %s:%d\n", hipGetErrorString(_e), __FILE__, __LINE__); \
    return; } } while (0)

namespace {

#define LAS __attribute__((address_space(3)))
typedef int v2i __attribute__((ext_vector_type(2)));
typedef int v4i __attribute__((ext_vector_type(4)));
typedef float v4f __attribute__((ext_vector_type(4)));
typedef unsigned v4u __attribute__((ext_vector_type(4)));

constexpr int NR = 8;
constexpr int KTOT = 4096, NFULL = 16384, NLOC = 2048, MLOC = 512;
constexpr int RING = 6, PF = 4;
constexpr int LPS = 3;
constexpr int STAGE_BYTES = 24576;
constexpr int CTL_OFF = RING * STAGE_BYTES;
constexpr int CNT_OFF = CTL_OFF + 64;
constexpr int LDS_BYTES = CNT_OFF + 256;
constexpr int IMG_BYTES = 256 * 64;

constexpr size_t WS_TICKET = 0;
constexpr size_t WS_TMO = 2048;
constexpr size_t WS_FLAG = 4096;
constexpr size_t WS_DUMMY = 128u << 10;
constexpr size_t WS_X = 1u << 20;
constexpr size_t SHARD_BYTES = (size_t)MLOC * KTOT;
constexpr size_t WS_NEED = WS_X + 2 * NR * SHARD_BYTES;

constexpr unsigned long long DEADLINE = 50000000ull;
constexpr int NCHUNK = 64;
#define PUSH_W 2
constexpr int PW = PUSH_W;
#define LOOKAHEAD 4
#define SKEW_A 0
constexpr unsigned TICKETS = 44;

struct Params {
    const char* x; const char* w; const float* sx; const float* sw; float* out;
    char* ws[NR];
    int rank;
};
#define STAMP(i) do {} while (0)

template <int V> struct IC { static constexpr int value = V; };
template <int N, int I = 0, class F> __device__ __forceinline__ void static_for(F&& f) { if constexpr (I < N) { f(IC<I>{}); static_for<N, I + 1>(f); } }
template <int N> __device__ __forceinline__ void waitv() { asm volatile("s_waitcnt vmcnt(%0)" :: "i"(N) : "memory"); }
__device__ __forceinline__ void bar() { asm volatile("" ::: "memory"); __builtin_amdgcn_s_barrier(); asm volatile("" ::: "memory"); }

__device__ __forceinline__ void lds_dma(unsigned voff, const char* base, unsigned ldst) {
    asm volatile("s_mov_b32 m0, %2\n\ts_nop 0\n\tglobal_load_lds_dwordx4 %0, %1" :: "v"(voff), "s"(base), "s"(__builtin_amdgcn_readfirstlane(ldst)) : "memory");
}

__device__ __forceinline__ const char* uniform_ptr(const char* q) {
    const unsigned long long v = (unsigned long long)q;
    const char* r = (const char*)(((unsigned long long)(unsigned)__builtin_amdgcn_readfirstlane((unsigned)(v >> 32)) << 32) |
                                  (unsigned)__builtin_amdgcn_readfirstlane((unsigned)v));
    asm volatile("s_nop 4" : "+s"(r));
    return r;
}

constexpr int SA = SKEW_A, SB = SA - 1, SBLK = SA ? 32 / SA : 0, SHEAD = SBLK * (SA + SB);
__device__ __forceinline__ int order_rb(int i) { if constexpr (SA == 0) return i & 1; else return i < SHEAD ? (i % (SA + SB)) & 1 : 1; }
__device__ __forceinline__ int order_kt(int i) {
    if constexpr (SA == 0) return i >> 1;
    else { if (i >= SHEAD) return SBLK * SB + (i - SHEAD);
           const int blk = i / (SA + SB), r = i % (SA + SB); return ((r & 1) ? SB : SA) * blk + (r >> 1); } }
__device__ __forceinline__ int order_pos(int rb, int kt) {
    if constexpr (SA == 0) return 2 * kt + rb;
    else { if (rb == 0) return (SA + SB) * (kt / SA) + 2 * (kt % SA);
           return kt < SBLK * SB ? (SA + SB) * (kt / SB) + 2 * (kt % SB) + 1 : SHEAD + kt - SBLK * SB; } }

__device__ __forceinline__ float silu_dq(int a, float s) {
    const float y = (float)a * s;
    const float c = __builtin_amdgcn_fmed3f(y, -60.f, 60.f);
    const float e = __builtin_amdgcn_exp2f(c * -1.44269504088896341f);
    return y * __builtin_amdgcn_rcpf(1.0f + e);
}

#define OUT_ST 2
__device__ __forceinline__ void store_out(float* q, v4f o) {
    if (OUT_ST == 0) *(v4f*)q = o;
    else if (OUT_ST == 1) __builtin_nontemporal_store(o, (v4f*)q);
    else if (OUT_ST == 2) asm volatile("global_store_dwordx4 %0, %1, off sc1\n\ts_nop 1" :: "v"(q), "v"(o) : "memory");
    else if (OUT_ST == 3) asm volatile("global_store_dwordx4 %0, %1, off sc0 sc1\n\ts_nop 1" :: "v"(q), "v"(o) : "memory");
    else asm volatile("global_store_dwordx4 %0, %1, off sc1 nt\n\ts_nop 1" :: "v"(q), "v"(o) : "memory");
}

template <bool BIG> struct Cfg {
    static constexpr int MT = BIG ? 4 : 2, NT = BIG ? 4 : 2, WN = BIG ? 2 : 4, NSUB = BIG ? 1 : 2;
    static constexpr int NSTEP = BIG ? 64 : 32;
    static constexpr int A_BYTES = BIG ? 16384 : 8192, B_BYTES = BIG ? 8192 : 16384;
    static constexpr int A_OFF = BIG ? RING * 8192 : 0, B_OFF = BIG ? 0 : RING * 8192;
    static constexpr int NBAR = 2 * NSTEP + 2;
};

template <bool BIG>
__device__ __forceinline__ void gemm_tile(LAS char* lds, int wid, int lane, const char* ab, const char* wb, float* out, float scale) {
    using C = Cfg<BIG>;
    constexpr int MT = C::MT, NT = C::NT, WN = C::WN, NSUB = C::NSUB, A_BYTES = C::A_BYTES, B_BYTES = C::B_BYTES;
    constexpr int NMAIN = C::NSTEP - PF, NLOOP = NMAIN / RING * RING;
    const unsigned lds0 = __builtin_amdgcn_readfirstlane((unsigned)(unsigned long long)lds);
    ab = uniform_ptr(ab); wb = uniform_ptr(wb);
    const int set = wid >> 2;
    const int wm = wid / WN, wn = wid % WN;
    const int li = lane & 15, q = lane >> 4;
    unsigned voff0, voff1, voff2, d0, d1, d2;
    LAS char* ra; LAS char* rb_[NT];
    const int fb = ((li >> 2) & 3) | ((q & 1) << 2);
    auto wswz = [&](int k) { return (unsigned)(k * NFULL + (((lane & 7) ^ (((k >> 1) & 3) | (((k >> 4) & 1) << 2))) * 16)); };
    if constexpr (BIG) {
        voff0 = (unsigned)(wid * 2048 + lane * 16); voff1 = voff0 + 1024; voff2 = wswz(wid * 8 + (lane >> 3));
        d0 = lds0 + C::A_OFF + wid * 2048; d1 = d0 + 1024; d2 = lds0 + C::B_OFF + wid * 1024;
        ra = lds + C::A_OFF + (unsigned)((wm * MT * 16 + li) * 64 + ((q ^ ((-(li >> 2)) & 3)) * 16));
    } else {
        const int r = wid * 8 + (lane >> 3);
        voff0 = (unsigned)(r * KTOT + (((lane & 7) ^ ((r >> 1) & 7)) * 16));
        voff1 = wswz(wid * 16 + (lane >> 3)); voff2 = wswz(wid * 16 + 8 + (lane >> 3));
        d0 = lds0 + C::A_OFF + wid * 1024; d1 = lds0 + C::B_OFF + wid * 2048; d2 = d1 + 1024;
        ra = lds + C::A_OFF + (unsigned)((wm * MT * 16 + li) * 128 + ((q ^ ((li >> 1) & 7)) * 16));
    }
#pragma unroll
    for (int n = 0; n < NT; ++n) rb_[n] = lds + C::B_OFF + (unsigned)((16 * q + (li >> 1)) * 128 + (((wn * NT + n) ^ fb) * 16) + 8 * (li & 1));
    v4i acc[MT][NT];
#pragma unroll
    for (int m = 0; m < MT; ++m)
#pragma unroll
        for (int n = 0; n < NT; ++n) acc[m][n] = (v4i){0, 0, 0, 0};

    auto issue = [&](int sl) {
        if constexpr (BIG) {
            lds_dma(voff0, ab, d0 + (unsigned)(sl * A_BYTES)); lds_dma(voff1, ab, d1 + (unsigned)(sl * A_BYTES)); lds_dma(voff2, wb, d2 + (unsigned)(sl * B_BYTES));
            ab += IMG_BYTES; wb += 64 * NFULL;
        } else {
            lds_dma(voff0, ab, d0 + (unsigned)(sl * A_BYTES)); lds_dma(voff1, wb, d1 + (unsigned)(sl * B_BYTES)); lds_dma(voff2, wb, d2 + (unsigned)(sl * B_BYTES));
            ab += 128; wb += 128 * NFULL;
        }
    };
    auto step = [&](auto slotc, auto islotc, auto wvc) {
        constexpr int sl = decltype(slotc)::value, isl = decltype(islotc)::value, WV = decltype(wvc)::value;
        v4i wf[NSUB][NT], xf[NSUB][MT];
#pragma unroll
        for (int s = 0; s < NSUB; ++s) {
#pragma unroll
            for (int n = 0; n < NT; ++n) {
                const v2i lo = __builtin_amdgcn_ds_read_tr8_b64_v2i32((LAS v2i*)(rb_[n] + sl * B_BYTES + s * 8192));
                const v2i hi = __builtin_amdgcn_ds_read_tr8_b64_v2i32((LAS v2i*)(rb_[n] + sl * B_BYTES + s * 8192 + 1024));
                wf[s][n] = (v4i){lo.x, lo.y, hi.x, hi.y};
            }
#pragma unroll
            for (int m = 0; m < MT; ++m) {
                if constexpr (BIG) xf[s][m] = *(LAS v4i*)(ra + sl * A_BYTES + m * 1024);
                else xf[s][m] = *(LAS v4i*)((LAS char*)((unsigned)(unsigned long long)ra ^ (unsigned)(s * 64)) + sl * A_BYTES + m * 2048);
            }
        }
        __builtin_amdgcn_sched_barrier(0);
        if constexpr (isl >= 0) issue(isl);
        if constexpr (WV >= 0) waitv<WV>();
        bar();
        asm volatile("s_waitcnt lgkmcnt(0)" ::: "memory");
        __builtin_amdgcn_sched_barrier(0);
        __builtin_amdgcn_s_setprio(1);
#pragma unroll
        for (int s = 0; s < NSUB; ++s)
#pragma unroll
            for (int m = 0; m < MT; ++m)
#pragma unroll
                for (int n = 0; n < NT; ++n) acc[m][n] = __builtin_amdgcn_mfma_i32_16x16x64_i8(wf[s][n], xf[s][m], acc[m][n], 0, 0, 0);
        __builtin_amdgcn_s_setprio(0);
        __builtin_amdgcn_sched_barrier(0);
        bar();
    };
#pragma unroll
    for (int s = 0; s < PF; ++s) issue(s);
    waitv<(PF - 1) * LPS>();
    bar();
    if (set == 1) bar();
    for (int j = 0; j < NLOOP; j += RING)
        static_for<RING>([&](auto ic) { constexpr int i = decltype(ic)::value; step(IC<i>{}, IC<(i + PF) % RING>{}, IC<(PF - 1) * LPS>{}); });
    static_for<NMAIN - NLOOP>([&](auto ic) { constexpr int i = decltype(ic)::value; step(IC<i % RING>{}, IC<(i + PF) % RING>{}, IC<(PF - 1) * LPS>{}); });
    static_for<PF>([&](auto ic) { constexpr int i = decltype(ic)::value;
        step(IC<(NMAIN + i) % RING>{}, IC<-1>{}, IC<(PF - 2 - i >= 0) ? (PF - 2 - i) * LPS : -1>{}); });
    if (set == 0) bar();
    const bool bad = ((volatile LAS unsigned*)(lds + CTL_OFF))[1] != 0u;
    const float qnan = __builtin_nanf("");
    float* op = out + (size_t)(wm * MT * 16 + li) * NLOC + wn * NT * 16 + 4 * q;
#pragma unroll
    for (int m = 0; m < MT; ++m)
#pragma unroll
        for (int n = 0; n < NT; ++n) {
            v4f o; o.x = silu_dq(acc[m][n].x, scale); o.y = silu_dq(acc[m][n].y, scale); o.z = silu_dq(acc[m][n].z, scale); o.w = silu_dq(acc[m][n].w, scale);
            if (bad) o = (v4f){qnan, qnan, qnan, qnan};
            store_out(op + (size_t)m * 16 * NLOC + n * 16, o);
        }
}

__global__ __launch_bounds__(576) void fused_kernel(Params p) {
    extern __shared__ __attribute__((aligned(16))) char shm[];
    LAS char* lds = (LAS char*)shm;
    volatile LAS unsigned* ctl = (volatile LAS unsigned*)(lds + CTL_OFF);
    const int tid = threadIdx.x, lane = tid & 63, wid = __builtin_amdgcn_readfirstlane(tid >> 6);
    const int b = blockIdx.x, grp = b & 7, slot = b >> 3;
    char* myws = p.ws[p.rank];

    if (slot < 4) {
        if (wid >= 4) return;
        unsigned tk = 0;
        if (lane == 0) tk = __hip_atomic_fetch_add((unsigned*)(myws + WS_TICKET + grp * 128), 1u, __ATOMIC_RELAXED, __HIP_MEMORY_SCOPE_AGENT);
        if (grp == 7) return;
        ctl[16 + tid % 64] = 0u;
        asm volatile("s_waitcnt lgkmcnt(0)" ::: "memory");
        const int gw = slot * 4 + wid;
        const int dstr = (p.rank + 1 + grp) & 7;
        const char* xs = p.x;
        const unsigned soff = (unsigned)((16 * gw + (lane >> 3)) * KTOT + (lane & 7) * 16);
        const unsigned dbase = (unsigned)(((lane & 7) >> 2) * IMG_BYTES + (16 * gw + (lane >> 3)) * 64);
        const unsigned doff0 = dbase + (unsigned)((((lane & 3) ^ ((-(lane >> 5)) & 3)) * 16));
        const unsigned doff1 = dbase + (unsigned)(8 * 64 + (((lane & 3) ^ ((-(2 + (lane >> 5))) & 3)) * 16));
        v4u R[PW][2];
        auto ld = [&](v4u (&r)[2], int c) {
            const int cc = c < NCHUNK ? c : NCHUNK - 1;
            const char* sp = xs + (unsigned)(order_rb(cc) * 256 * KTOT + order_kt(cc) * 128);
            asm volatile("global_load_dwordx4 %0, %1, %2" : "=v"(r[0]) : "v"(soff), "s"(sp) : "memory");
            asm volatile("global_load_dwordx4 %0, %1, %2" : "=v"(r[1]) : "v"(soff + 8u * KTOT), "s"(sp) : "memory"); };
#pragma unroll
        for (int i = 0; i < PW; ++i) ld(R[i], i);
        tk = __builtin_amdgcn_readfirstlane(tk);
        waitv<0>();
        bar();
        const unsigned call = tk / TICKETS + 1u;
        char* dst = p.ws[dstr] + WS_X + ((size_t)(call & 1u) * NR + p.rank) * SHARD_BYTES;
        char* flag = p.ws[dstr] + WS_FLAG + (size_t)p.rank * NCHUNK * 128 + slot * 4;
        const unsigned cnt0 = (unsigned)(unsigned long long)lds + CNT_OFF;
        auto publish = [&](int t) {
            unsigned old = 0;
            if (t >= 0) {
                if (lane == 0) asm volatile("ds_add_rtn_u32 %0, %1, %2\n\ts_waitcnt lgkmcnt(0)" : "=&v"(old) : "v"(cnt0 + (unsigned)t * 4u), "v"(1u) : "memory");
                old = __builtin_amdgcn_readfirstlane(old);
            }
            char* fp = (t >= 0 && old == 3u) ? flag + (size_t)t * 128 : myws + WS_DUMMY + (b * 4 + wid) * 128;
            if (lane == 0) asm volatile("global_store_dword %0, %1, off sc0 sc1" :: "v"(fp), "v"(call) : "memory"); };
        for (int c0 = 0; c0 < NCHUNK; c0 += PW) {
            static_for<PW>([&](auto ic) {
                constexpr int i = decltype(ic)::value;
                const int c = c0 + i;
                if (c < NCHUNK) {
                    waitv<5 * (PW - 1)>();
                    publish(c - PW);
                    const char* dp = dst + (unsigned)(order_rb(c) * (256 * KTOT) + order_kt(c) * 2 * IMG_BYTES);
                    asm volatile("" : "+v"(R[i][0]), "+v"(R[i][1]));
                    asm volatile("global_store_dwordx4 %0, %1, %2 sc0 sc1\n\ts_nop 1" :: "v"(doff0), "v"(R[i][0]), "s"(dp) : "memory");
                    asm volatile("global_store_dwordx4 %0, %1, %2 sc0 sc1\n\ts_nop 1" :: "v"(doff1), "v"(R[i][1]), "s"(dp) : "memory");
                    ld(R[i], c + PW);
                }
            });
        }
        static_for<PW>([&](auto ic) {
            constexpr int v = decltype(ic)::value;
            waitv<2 + 5 * (PW - 1 - v) + v>();
            publish(NCHUNK - PW + v);
        });
        waitv<0>();
#pragma unroll
        for (int i = 0; i < PW; ++i) asm volatile("" :: "v"(R[i][0]), "v"(R[i][1]));
        if (wid == 0) STAMP(7);
        return;
    }

    const int j = slot - 4, pj = j >> 2;
    const int rb = ((grp >> 2) ^ pj) & 1;
    const int src = (p.rank + 1 + pj) & 7;
    const int cb = (grp & 3) * 4 + (j & 3);
    const bool has_own = pj < 4;

    if (wid == 8) {
        unsigned tk = 0;
        if (lane == 0) tk = __hip_atomic_fetch_add((unsigned*)(myws + WS_TICKET + grp * 128), 1u, __ATOMIC_RELAXED, __HIP_MEMORY_SCOPE_AGENT);
        tk = __builtin_amdgcn_readfirstlane(tk);
        const unsigned call = tk / TICKETS + 1u;
        const char* flag = myws + WS_FLAG + (size_t)src * NCHUNK * 128 + (lane & 3) * 4;
        unsigned* tmo = (unsigned*)(myws + WS_TMO);
        int ready = 0;
        bool bad = false;
        auto ensure = [&](int step) {
            const int need = (step >> 1) + 1;
            if (ready >= need) return;
            const unsigned long long t0 = __builtin_amdgcn_s_memrealtime();
            for (;;) {
                int kt = ready + ((lane >> 2) & (LOOKAHEAD - 1)); kt = kt < 32 ? kt : 31;
                const unsigned v = __hip_atomic_load((const unsigned*)(flag + order_pos(rb, kt) * 128), __ATOMIC_RELAXED, __HIP_MEMORY_SCOPE_SYSTEM);
                unsigned long long m = __ballot(v >= call);
                m = m & (m >> 1); m = m & (m >> 2);
                int adv = 0;
#pragma unroll
                for (int k = 0; k < LOOKAHEAD; ++k) { if (adv == k && ((m >> (4 * k)) & 1ull)) adv = k + 1; }
                ready += adv; ready = ready < 32 ? ready : 32;
                if (ready >= need) break;
                const unsigned tv = __builtin_amdgcn_readfirstlane(__hip_atomic_load(tmo, __ATOMIC_RELAXED, __HIP_MEMORY_SCOPE_AGENT));
                if (tv == call || __builtin_amdgcn_s_memrealtime() - t0 > DEADLINE) {
                    if (lane == 0) __hip_atomic_store(tmo, call, __ATOMIC_RELAXED, __HIP_MEMORY_SCOPE_AGENT);
                    bad = true; ready = 64; break;
                }
                __builtin_amdgcn_s_sleep(2);
            }
        };
        if (lane == 0) { ctl[0] = call; ctl[1] = 0u; }
        asm volatile("s_waitcnt lgkmcnt(0)" ::: "memory");
        if (has_own) for (int i = 0; i < Cfg<false>::NBAR; ++i) bar();
        ensure(PF - 1);
        ensure(PF);
        bar(); bar();
        for (int s = 0; s < 64; ++s) {
            bar();
            if (s + 1 + PF < 64) ensure(s + 1 + PF);
            if (s == 63) { if (lane == 0) ctl[1] = bad ? 1u : 0u; asm volatile("s_waitcnt lgkmcnt(0)" ::: "memory"); }
            bar();
        }
        bar();
        STAMP(6);
        return;
    }

    const float scale = p.sx[0] * p.sw[0];
    const char* wb = p.w + (size_t)p.rank * NLOC + cb * 128;
    if (has_own) {
        const int row0 = ((grp >> 2) * 4 + pj) * 64;
        gemm_tile<false>(lds, wid, lane, p.x + (size_t)row0 * KTOT, wb, p.out + (size_t)(p.rank * MLOC + row0) * NLOC + cb * 128, scale);
        if (wid == 0) STAMP(0);
    }
    bar();
    const unsigned call = ctl[0];
    const char* ab = p.ws[p.rank] + WS_X + ((size_t)(call & 1u) * NR + src) * SHARD_BYTES + (size_t)rb * 256 * KTOT;
    gemm_tile<true>(lds, wid, lane, ab, wb, p.out + (size_t)(src * MLOC + rb * 256) * NLOC + cb * 128, scale);
}

}

extern "C" void kernel_launch(int rank, int nranks, ncclComm_t comm,
                              hipStream_t stream,
                              void* const* d_in, const int* in_sizes, int n_in,
                              void* d_out, int out_size,
                              const int* in_dtypes, int out_dtype,
                              void* const* d_out_all,
                              void* const* d_ws_all, size_t ws_bytes) {
    (void)comm; (void)in_sizes; (void)n_in; (void)out_size; (void)in_dtypes; (void)out_dtype; (void)d_out_all;
    if (nranks != NR || rank < 0 || rank >= NR || ws_bytes < WS_NEED) { fprintf(stderr, "unexpected configuration\n"); return; }
    HIP_CHECK(hipFuncSetAttribute((const void*)fused_kernel, hipFuncAttributeMaxDynamicSharedMemorySize, LDS_BYTES));
    Params p;
    p.x = (const char*)d_in[0]; p.w = (const char*)d_in[1]; p.sx = (const float*)d_in[2]; p.sw = (const float*)d_in[3];
    p.out = (float*)d_out; p.rank = rank;
    for (int i = 0; i < NR; ++i) p.ws[i] = (char*)d_ws_all[i];
    hipLaunchKernelGGL(fused_kernel, dim3(256), dim3(576), LDS_BYTES, stream, p);
    HIP_CHECK(hipGetLastError());
}
```

```cpp
#include <hip/hip_runtime.h>
#include <rccl/rccl.h>
#include <cstdio>

#define HIP_CHECK(expr) do { hipError_t _e = (expr); if (_e != hipSuccess) { \
    fprintf(stderr, "HIP error %s at %s:%d\n", hipGetErrorString(_e), __FILE__, __LINE__); \
    return; } } while (0)

namespace {

#define LAS __attribute__((address_space(3)))
typedef int v2i __attribute__((ext_vector_type(2)));
typedef int v4i __attribute__((ext_vector_type(4)));
typedef float v4f __attribute__((ext_vector_type(4)));
typedef unsigned v4u __attribute__((ext_vector_type(4)));

constexpr int NR = 8;
constexpr int KTOT = 4096, NFULL = 16384, NLOC = 2048, MLOC = 512;
constexpr int RING_BYTES = 147456;
constexpr int CTL_OFF = RING_BYTES;
constexpr int CNT_OFF = CTL_OFF + 64;
constexpr int LDS_BYTES = CNT_OFF + 256;
constexpr int IMG_BYTES = 16384, RB_BYTES = 64 * IMG_BYTES;

constexpr size_t WS_TICKET = 0;
constexpr size_t WS_TMO = 2048;
constexpr size_t WS_FLAG = 4096;
constexpr size_t WS_DUMMY = 128u << 10;
constexpr size_t WS_X = 1u << 20;
constexpr size_t SHARD_BYTES = (size_t)MLOC * KTOT;
constexpr size_t WS_NEED = WS_X + 2 * NR * SHARD_BYTES;

constexpr unsigned long long DEADLINE = 50000000ull;
constexpr int NCHUNK = 64;
#define PUSH_W 2
constexpr int PW = PUSH_W;
#define LOOKAHEAD 16
#define SKEW_A 0
#define PF_BIG 3
#define WARM 7
constexpr int CODE_WARM = 32768;
constexpr int WARM_N = 10 + 5 + 1 + CODE_WARM / 8192;
constexpr unsigned TICKETS = 44;

struct Params {
    const char* x; const char* w; const float* sx; const float* sw; float* out;
    char* ws[NR];
    int rank;
};
#define STAMP(i) do {} while (0)

template <int V> struct IC { static constexpr int value = V; };
template <int N, int I = 0, class F> __device__ __forceinline__ void static_for(F&& f) { if constexpr (I < N) { f(IC<I>{}); static_for<N, I + 1>(f); } }
template <int N> __device__ __forceinline__ void waitv() { asm volatile("s_waitcnt vmcnt(%0)" :: "i"(N) : "memory"); }
__device__ __forceinline__ void bar() { asm volatile("" ::: "memory"); __builtin_amdgcn_s_barrier(); asm volatile("" ::: "memory"); }

__device__ __forceinline__ void lds_dma(unsigned voff, const char* base, unsigned ldst) {
    asm volatile("s_mov_b32 m0, %2\n\ts_nop 0\n\tglobal_load_lds_dwordx4 %0, %1" :: "v"(voff), "s"(base), "s"(__builtin_amdgcn_readfirstlane(ldst)) : "memory");
}

__device__ __forceinline__ const char* uniform_ptr(const char* q) {
    const unsigned long long v = (unsigned long long)q;
    const char* r = (const char*)(((unsigned long long)(unsigned)__builtin_amdgcn_readfirstlane((unsigned)(v >> 32)) << 32) |
                                  (unsigned)__builtin_amdgcn_readfirstlane((unsigned)v));
    asm volatile("s_nop 4" : "+s"(r));
    return r;
}

constexpr int SA = SKEW_A, SB_ = SA - 1, SBLK = SA ? 32 / SA : 0, SHEAD = SBLK * (SA + SB_);
__device__ __forceinline__ int order_rb(int i) { if constexpr (SA == 0) return i & 1; else return i < SHEAD ? (i % (SA + SB_)) & 1 : 1; }
__device__ __forceinline__ int order_kt(int i) {
    if constexpr (SA == 0) return i >> 1;
    else { if (i >= SHEAD) return SBLK * SB_ + (i - SHEAD);
           const int blk = i / (SA + SB_), r = i % (SA + SB_); return ((r & 1) ? SB_ : SA) * blk + (r >> 1); } }
__device__ __forceinline__ int order_pos(int rb, int kt) {
    if constexpr (SA == 0) return 2 * kt + rb;
    else { if (rb == 0) return (SA + SB_) * (kt / SA) + 2 * (kt % SA);
           return kt < SBLK * SB_ ? (SA + SB_) * (kt / SB_) + 2 * (kt % SB_) + 1 : SHEAD + kt - SBLK * SB_; } }

__device__ __forceinline__ float silu_dq(int a, float s) {
    const float y = (float)a * s;
    const float c = __builtin_amdgcn_fmed3f(y, -60.f, 60.f);
    const float e = __builtin_amdgcn_exp2f(c * -1.44269504088896341f);
    return y * __builtin_amdgcn_rcpf(1.0f + e);
}

#define OUT_ST 2
__device__ __forceinline__ void store_out(float* q, v4f o) {
    if (OUT_ST == 0) *(v4f*)q = o;
    else if (OUT_ST == 1) __builtin_nontemporal_store(o, (v4f*)q);
    else if (OUT_ST == 2) asm volatile("global_store_dwordx4 %0, %1, off sc1\n\ts_nop 1" :: "v"(q), "v"(o) : "memory");
    else if (OUT_ST == 3) asm volatile("global_store_dwordx4 %0, %1, off sc0 sc1\n\ts_nop 1" :: "v"(q), "v"(o) : "memory");
    else asm volatile("global_store_dwordx4 %0, %1, off sc1 nt\n\ts_nop 1" :: "v"(q), "v"(o) : "memory");
}

template <bool BIG> struct Cfg {
    static constexpr int MT = BIG ? 4 : 2, NT = BIG ? 4 : 2, WN = BIG ? 2 : 4, NSUB = BIG ? 1 : 2;
    static constexpr int NSTEP = BIG ? 64 : 32;
    static constexpr int RING = 6, PF = BIG ? PF_BIG : 4;
    static constexpr int A_BYTES = BIG ? 16384 : 8192, B_BYTES = BIG ? 8192 : 16384;
    static constexpr int A_OFF = BIG ? RING * 8192 : 0, B_OFF = BIG ? 0 : RING * 8192;
    static constexpr int L = 3;
    static constexpr int NBAR = 2 * NSTEP + 2;
    static_assert(RING * (A_BYTES + B_BYTES) <= RING_BYTES && PF <= RING - 1, "ring");
};

template <bool BIG, int SET>
__device__ __forceinline__ void gemm_tile_set(LAS char* lds, int wid, int lane, const char* ab, unsigned a_voff0, unsigned a_voff1, int a_step,
                                              const char* wb, float* out, float scale) {
    using C = Cfg<BIG>;
    constexpr int MT = C::MT, NT = C::NT, WN = C::WN, NSUB = C::NSUB, A_BYTES = C::A_BYTES, B_BYTES = C::B_BYTES, RING = C::RING, PF = C::PF, L = C::L;
    constexpr int NMAIN = C::NSTEP - PF, NLOOP = NMAIN / RING * RING;
    const unsigned lds0 = __builtin_amdgcn_readfirstlane((unsigned)(unsigned long long)lds);
    ab = uniform_ptr(ab); wb = uniform_ptr(wb);
    wid = __builtin_amdgcn_readfirstlane(wid); a_step = __builtin_amdgcn_readfirstlane(a_step);
    const int wm = wid / WN, wn = wid % WN;
    const int li = lane & 15, q = lane >> 4;
    auto wswz = [&](int k) { return (unsigned)(k * NFULL + (((lane & 7) ^ (((k >> 1) & 3) | (((k >> 4) & 1) << 2))) * 16)); };
    const unsigned b_voff0 = wswz((BIG ? wid * 8 : wid * 16) + (lane >> 3)), b_voff1 = wswz(wid * 16 + 8 + (lane >> 3));
    unsigned dA = lds0 + C::A_OFF + wid * (BIG ? 2048 : 1024), dB = lds0 + C::B_OFF + wid * (BIG ? 1024 : 2048);
    asm volatile("" : "+s"(dA), "+s"(dB));
    LAS char* ra0 = lds + C::A_OFF + (BIG ? (unsigned)((wm * MT * 16 + li) * 64 + ((q ^ ((-(li >> 2)) & 3)) * 16))
                                          : (unsigned)((wm * MT * 16 + li) * 128 + ((q ^ ((li >> 1) & 7)) * 16)));
    LAS char* ra1 = ra0 + (((q ^ ((li >> 1) & 7)) & 4) ? -64 : 64);
    LAS char* rb_[NT];
    const int fb = ((li >> 2) & 3) | ((q & 1) << 2);
#pragma unroll
    for (int n = 0; n < NT; ++n) rb_[n] = lds + C::B_OFF + (unsigned)((16 * q + (li >> 1)) * 128 + (((wn * NT + n) ^ fb) * 16) + 8 * (li & 1));
    v4i acc[MT][NT];
#pragma unroll
    for (int m = 0; m < MT; ++m)
#pragma unroll
        for (int n = 0; n < NT; ++n) acc[m][n] = (v4i){0, 0, 0, 0};

    auto dma = [&](auto ic, int sl) {
        constexpr int i = decltype(ic)::value;
        if constexpr (i == 0) lds_dma(a_voff0, ab, dA + (unsigned)(sl * A_BYTES));
        if constexpr (i == 1 && BIG) lds_dma(a_voff1, ab, dA + (unsigned)(sl * A_BYTES + 1024));
        if constexpr (i == 1 && !BIG) lds_dma(b_voff0, wb, dB + (unsigned)(sl * B_BYTES));
        if constexpr (i == 2 && BIG) lds_dma(b_voff0, wb, dB + (unsigned)(sl * B_BYTES));
        if constexpr (i == 2 && !BIG) lds_dma(b_voff1, wb, dB + (unsigned)(sl * B_BYTES + 1024));
        if constexpr (i == L - 1) { ab += a_step; wb += (BIG ? 64 : 128) * NFULL; }
    };
    auto issue = [&](int sl) { static_for<L>([&](auto ic) { dma(ic, sl); }); };
    auto step = [&](auto slotc, auto islotc, auto wvc) {
        constexpr int sl = decltype(slotc)::value, isl = decltype(islotc)::value, WV = decltype(wvc)::value;
        v4i wf[NSUB][NT], xf[NSUB][MT];
        auto rdw = [&](int s, int n) {
            const v2i lo = __builtin_amdgcn_ds_read_tr8_b64_v2i32((LAS v2i*)(rb_[n] + sl * B_BYTES + s * 8192));
            const v2i hi = __builtin_amdgcn_ds_read_tr8_b64_v2i32((LAS v2i*)(rb_[n] + sl * B_BYTES + s * 8192 + 1024));
            wf[s][n] = (v4i){lo.x, lo.y, hi.x, hi.y}; };
        auto rdx = [&](int s, int m) { xf[s][m] = *(LAS v4i*)((s ? ra1 : ra0) + sl * A_BYTES + m * (BIG ? 1024 : 2048)); };
        auto idma = [&](auto ic) { __builtin_amdgcn_sched_barrier(0); if constexpr (isl >= 0) dma(ic, isl); __builtin_amdgcn_sched_barrier(0); };
        if constexpr (BIG) {
            rdw(0, 0); rdw(0, 1); idma(IC<0>{});
            rdw(0, 2); rdw(0, 3); idma(IC<1>{});
            rdx(0, 0); rdx(0, 1); rdx(0, 2); rdx(0, 3); idma(IC<2>{});
        } else {
            rdw(0, 0); rdw(0, 1); idma(IC<0>{});
            rdx(0, 0); rdx(0, 1); rdw(1, 0); idma(IC<1>{});
            rdw(1, 1); rdx(1, 0); rdx(1, 1); idma(IC<2>{});
        }
        if constexpr (WV >= 0) waitv<WV * L>();
        asm volatile("s_waitcnt lgkmcnt(0)" ::: "memory");
        bar();
        __builtin_amdgcn_sched_barrier(0);
        __builtin_amdgcn_s_setprio(1);
#pragma unroll
        for (int s = 0; s < NSUB; ++s)
#pragma unroll
            for (int m = 0; m < MT; ++m)
#pragma unroll
                for (int n = 0; n < NT; ++n) acc[m][n] = __builtin_amdgcn_mfma_i32_16x16x64_i8(wf[s][n], xf[s][m], acc[m][n], 0, 0, 0);
        __builtin_amdgcn_s_setprio(0);
        __builtin_amdgcn_sched_barrier(0);
        bar();
    };
#pragma unroll
    for (int s = 0; s < PF; ++s) issue(s);
    waitv<(PF - 1) * L>();
    bar();
    if constexpr (SET == 1) bar();
    for (int j = 0; j < NLOOP; j += RING)
        static_for<RING>([&](auto ic) { constexpr int i = decltype(ic)::value; step(IC<i>{}, IC<(i + PF) % RING>{}, IC<PF - 1>{}); });
    static_for<NMAIN - NLOOP>([&](auto ic) { constexpr int i = decltype(ic)::value; step(IC<i % RING>{}, IC<(i + PF) % RING>{}, IC<PF - 1>{}); });
    static_for<PF>([&](auto ic) { constexpr int i = decltype(ic)::value;
        step(IC<(NMAIN + i) % RING>{}, IC<-1>{}, IC<PF - 2 - i>{}); });
    if constexpr (SET == 0) bar();
    const bool bad = __builtin_amdgcn_readfirstlane(((volatile LAS unsigned*)(lds + CTL_OFF))[1]) != 0u;
    float* op = out + (size_t)(wm * MT * 16 + (li & 7)) * NLOC + wn * NT * 16 + (li >> 3) * 16 + 4 * q;
    if (!bad) {
#pragma unroll
        for (int m = 0; m < MT; ++m)
#pragma unroll
            for (int n = 0; n < NT; n += 2) {
                v4i lo, hi;
#pragma unroll
                for (int e = 0; e < 4; ++e) {
                    lo[e] = __builtin_amdgcn_update_dpp(acc[m][n][e], acc[m][n + 1][e], 0x128, 0xf, 0xc, false);
                    hi[e] = __builtin_amdgcn_update_dpp(acc[m][n + 1][e], acc[m][n][e], 0x128, 0xf, 0x3, false);
                }
                v4f o; o.x = silu_dq(lo.x, scale); o.y = silu_dq(lo.y, scale); o.z = silu_dq(lo.z, scale); o.w = silu_dq(lo.w, scale);
                store_out(op + (size_t)m * 16 * NLOC + n * 16, o);
                o.x = silu_dq(hi.x, scale); o.y = silu_dq(hi.y, scale); o.z = silu_dq(hi.z, scale); o.w = silu_dq(hi.w, scale);
                store_out(op + (size_t)(m * 16 + 8) * NLOC + n * 16, o);
            }
    } else {
        const float qnan = __builtin_nanf("");
        for (int m = 0; m < 2 * MT; ++m)
            for (int n = 0; n < NT; n += 2) store_out(op + (size_t)m * 8 * NLOC + n * 16, (v4f){qnan, qnan, qnan, qnan});
    }
}
template <bool BIG>
__device__ __forceinline__ void gemm_tile(LAS char* lds, int wid, int lane, const char* ab, unsigned a_voff0, unsigned a_voff1, int a_step,
                                          const char* wb, float* out, float scale) {
    wid = __builtin_amdgcn_readfirstlane(wid);
    if (wid < 4) gemm_tile_set<BIG, 0>(lds, wid, lane, ab, a_voff0, a_voff1, a_step, wb, out, scale);
    else gemm_tile_set<BIG, 1>(lds, wid, lane, ab, a_voff0, a_voff1, a_step, wb, out, scale);
}

__global__ __launch_bounds__(576) void fused_kernel(Params p) {
    extern __shared__ __attribute__((aligned(16))) char shm[];
    LAS char* lds = (LAS char*)shm;
    volatile LAS unsigned* ctl = (volatile LAS unsigned*)(lds + CTL_OFF);
    const int tid = threadIdx.x, lane = tid & 63, wid = __builtin_amdgcn_readfirstlane(tid >> 6);
    const int b = blockIdx.x, grp = b & 7, slot = b >> 3;
    char* myws = p.ws[p.rank];
    unsigned long long pc0;
    asm volatile("s_getpc_b64 %0" : "=s"(pc0));
    const char* code0 = (const char*)(pc0 & ~127ull);

    if (slot < 4) {
        if (wid >= 4) return;
        unsigned tk = 0;
        if (lane == 0) tk = __hip_atomic_fetch_add((unsigned*)(myws + WS_TICKET + grp * 128), 1u, __ATOMIC_RELAXED, __HIP_MEMORY_SCOPE_AGENT);
        if (grp == 7) return;
        ctl[16 + tid % 64] = 0u;
        asm volatile("s_waitcnt lgkmcnt(0)" ::: "memory");
        const int gw = slot * 4 + wid;
        const int dstr = (p.rank + 1 + grp) & 7;
        const char* xs = p.x;
        const int lr = lane >> 3, c8 = lane & 7;
        const unsigned soff = (unsigned)((16 * gw + lr) * KTOT + c8 * 16);
        unsigned doff[2];
#pragma unroll
        for (int i = 0; i < 2; ++i) { const int r = 16 * gw + 8 * i + lr;
            doff[i] = (unsigned)((c8 >> 2) * IMG_BYTES + r * 64 + (((c8 & 3) ^ ((-(r >> 2)) & 3)) * 16)); }
        v4u R[PW][2];
        auto ld = [&](v4u (&r)[2], int c) {
            const int cc = c < NCHUNK ? c : NCHUNK - 1;
            const char* sp = xs + (unsigned)(order_rb(cc) * 256 * KTOT + order_kt(cc) * 128);
            asm volatile("global_load_dwordx4 %0, %1, %2" : "=v"(r[0]) : "v"(soff), "s"(sp) : "memory");
            asm volatile("global_load_dwordx4 %0, %1, %2" : "=v"(r[1]) : "v"(soff + 8u * KTOT), "s"(sp) : "memory"); };
#pragma unroll
        for (int i = 0; i < PW; ++i) ld(R[i], i);
        tk = __builtin_amdgcn_readfirstlane(tk);
        waitv<0>();
        bar();
        const unsigned call = tk / TICKETS + 1u;
        char* dst = p.ws[dstr] + WS_X + ((size_t)(call & 1u) * NR + p.rank) * SHARD_BYTES;
        char* flag = p.ws[dstr] + WS_FLAG + (size_t)p.rank * NCHUNK * 128 + slot * 4;
        const unsigned cnt0 = (unsigned)(unsigned long long)lds + CNT_OFF;
        auto publish = [&](int t) {
            unsigned old = 0;
            if (t >= 0) {
                if (lane == 0) asm volatile("ds_add_rtn_u32 %0, %1, %2\n\ts_waitcnt lgkmcnt(0)" : "=&v"(old) : "v"(cnt0 + (unsigned)t * 4u), "v"(1u) : "memory");
                old = __builtin_amdgcn_readfirstlane(old);
            }
            char* fp = (t >= 0 && old == 3u) ? flag + (size_t)t * 128 : myws + WS_DUMMY + (b * 4 + wid) * 128;
            if (lane == 0) asm volatile("global_store_dword %0, %1, off sc0 sc1" :: "v"(fp), "v"(call) : "memory"); };
        static_assert(NCHUNK % PW == 0, "unroll");
        for (int c0 = 0; c0 < NCHUNK; c0 += PW) static_for<PW>([&](auto ic) {
            constexpr int i = decltype(ic)::value;
            const int c = c0 + i;
            waitv<5 * (PW - 1)>();
            publish(c - PW);
            const char* dp = dst + (unsigned)(order_rb(c) * RB_BYTES + order_kt(c) * 2 * IMG_BYTES);
            asm volatile("" : "+v"(R[i][0]), "+v"(R[i][1]));
            asm volatile("global_store_dwordx4 %0, %1, %2 sc0 sc1\n\ts_nop 1" :: "v"(doff[0]), "v"(R[i][0]), "s"(dp) : "memory");
            asm volatile("global_store_dwordx4 %0, %1, %2 sc0 sc1\n\ts_nop 1" :: "v"(doff[1]), "v"(R[i][1]), "s"(dp) : "memory");
            ld(R[i], c + PW);
        });
        static_for<PW>([&](auto ic) {
            constexpr int v = decltype(ic)::value;
            waitv<2 + 5 * (PW - 1 - v) + v>();
            publish(NCHUNK - PW + v);
        });
        waitv<0>();
#pragma unroll
        for (int i = 0; i < PW; ++i) asm volatile("" :: "v"(R[i][0]), "v"(R[i][1]));
        if (wid == 0) STAMP(7);
        return;
    }

    const int j = slot - 4, pj = j >> 2;
    const int h = grp >> 2;
    const int rb = (h ^ pj) & 1;
    const int src = (p.rank + 1 + pj) & 7;
    const int cb = (grp & 3) * 4 + (j & 3);
    const bool has_own = pj < 4;

    if (wid == 8) {
        unsigned tk = 0;
        if (lane == 0) tk = __hip_atomic_fetch_add((unsigned*)(myws + WS_TICKET + grp * 128), 1u, __ATOMIC_RELAXED, __HIP_MEMORY_SCOPE_AGENT);
        tk = __builtin_amdgcn_readfirstlane(tk);
        const unsigned call = tk / TICKETS + 1u;
        const char* flag = myws + WS_FLAG + (size_t)src * NCHUNK * 128 + (lane & 3) * 4;
        unsigned* tmo = (unsigned*)(myws + WS_TMO);
        int ready = 0;
        bool bad = false;
        auto poll = [&]() {
            int kt = ready + ((lane >> 2) & (LOOKAHEAD - 1)); kt = kt < 32 ? kt : 31;
            const unsigned v = __hip_atomic_load((const unsigned*)(flag + order_pos(rb, kt) * 128), __ATOMIC_RELAXED, __HIP_MEMORY_SCOPE_SYSTEM);
            unsigned long long m = __ballot(v >= call);
            m = m & (m >> 1); m = m & (m >> 2);
            m = ~m & 0x1111111111111111ull;
            if constexpr (LOOKAHEAD < 16) m |= 1ull << (4 * LOOKAHEAD);
            ready += m ? (int)(__builtin_ctzll(m) >> 2) : 16;
            ready = ready < 32 ? ready : 32;
        };
        auto ensure = [&](int kt) {
            const int need = kt + 1;
            if (ready >= need) return;
            const unsigned long long t0 = __builtin_amdgcn_s_memrealtime();
            for (;;) {
                poll();
                if (ready >= need) {
                    break; }
                const unsigned tv = __builtin_amdgcn_readfirstlane(__hip_atomic_load(tmo, __ATOMIC_RELAXED, __HIP_MEMORY_SCOPE_AGENT));
                if (tv == call || __builtin_amdgcn_s_memrealtime() - t0 > DEADLINE) {
                    if (lane == 0) __hip_atomic_store(tmo, call, __ATOMIC_RELAXED, __HIP_MEMORY_SCOPE_AGENT);
                    bad = true; ready = 32; break;
                }
                __builtin_amdgcn_s_sleep(2);
            }
        };
        if (lane == 0) { ctl[0] = call; ctl[1] = 0u; }
        asm volatile("s_waitcnt lgkmcnt(0)" ::: "memory");
        unsigned wr[WARM_N];
        {
            auto touch = [&](unsigned& r, const char* q) { asm volatile("global_load_dword %0, %1, off" : "=v"(r) : "v"(q) : "memory"); };
            int n = 0;
            const char* wq = p.w + (size_t)p.rank * NLOC + (grp & 3) * 512 + (size_t)(lane >> 2) * NFULL + (lane & 3) * 128;
#pragma unroll
            for (int i = 0; i < 10; ++i) { int t = j + 28 * i; t = t < 256 ? t : 255; touch(wr[n++], wq + (size_t)t * 16 * NFULL); }
            const char* xq = p.x + (size_t)h * 256 * KTOT + lane * 128;
#pragma unroll
            for (int i = 0; i < 5; ++i) { int t = j + 28 * i; t = t < 128 ? t : 127; touch(wr[n++], xq + (size_t)t * 8192); }
            touch(wr[n++], (const char*)(p.out + (size_t)(src * MLOC + rb * 256 + lane * 4) * NLOC + cb * 128));
            for (; n < WARM_N; ++n) wr[n] = 0;
        }
        if (has_own) for (int i = 0; i < Cfg<false>::NBAR; ++i) bar();
        waitv<0>();
#pragma unroll
        for (int i = 0; i < WARM_N; ++i) asm volatile("" :: "v"(wr[i]));
        constexpr int NS = Cfg<true>::NSTEP, PFB = Cfg<true>::PF;
        if (has_own) poll();
        ensure(PFB >> 1);
        bar(); bar();
        for (int s = 0; s < NS; ++s) {
            bar();
            if (s + 1 + PFB < NS) ensure((s + 1 + PFB) >> 1);
            if (s == NS - 1) { if (lane == 0) ctl[1] = bad ? 1u : 0u; asm volatile("s_waitcnt lgkmcnt(0)" ::: "memory"); }
            bar();
        }
        bar();
        STAMP(6);
        return;
    }

    const float scale = p.sx[0] * p.sw[0];
    const char* wb = p.w + (size_t)p.rank * NLOC + cb * 128;
    if (has_own) {
        const int row0 = (h * 4 + pj) * 64, r = wid * 8 + (lane >> 3);
        gemm_tile<false>(lds, wid, lane, p.x + (size_t)row0 * KTOT, (unsigned)(r * KTOT + (((lane & 7) ^ ((r >> 1) & 7)) * 16)), 0u, 128,
                         wb, p.out + (size_t)(p.rank * MLOC + row0) * NLOC + cb * 128, scale);
        if (wid == 0) STAMP(0);
    }
    bar();
    const char* xsrc = p.ws[p.rank] + WS_X + ((size_t)(ctl[0] & 1u) * NR + src) * SHARD_BYTES + (size_t)rb * RB_BYTES;
    const unsigned lin0 = (unsigned)(wid * 2048 + lane * 16);
    gemm_tile<true>(lds, wid, lane, xsrc, lin0, lin0 + 1024, IMG_BYTES, wb, p.out + (size_t)(src * MLOC + rb * 256) * NLOC + cb * 128, scale);
}

}

extern "C" void kernel_launch(int rank, int nranks, ncclComm_t comm,
                              hipStream_t stream,
                              void* const* d_in, const int* in_sizes, int n_in,
                              void* d_out, int out_size,
                              const int* in_dtypes, int out_dtype,
                              void* const* d_out_all,
                              void* const* d_ws_all, size_t ws_bytes) {
    (void)comm; (void)in_sizes; (void)n_in; (void)out_size; (void)in_dtypes; (void)out_dtype; (void)d_out_all;
    if (nranks != NR || rank < 0 || rank >= NR || ws_bytes < WS_NEED) { fprintf(stderr, "unexpected configuration\n"); return; }
    HIP_CHECK(hipFuncSetAttribute((const void*)fused_kernel, hipFuncAttributeMaxDynamicSharedMemorySize, LDS_BYTES));
    Params p;
    p.x = (const char*)d_in[0]; p.w = (const char*)d_in[1]; p.sx = (const float*)d_in[2]; p.sw = (const float*)d_in[3];
    p.out = (float*)d_out; p.rank = rank;
    for (int i = 0; i < NR; ++i) p.ws[i] = (char*)d_ws_all[i];
    hipLaunchKernelGGL(fused_kernel, dim3(256), dim3(576), LDS_BYTES, stream, p);
    HIP_CHECK(hipGetLastError());
}
```

```cpp
#include <hip/hip_runtime.h>
#include <rccl/rccl.h>
#include <cstdio>

#define HIP_CHECK(expr) do { hipError_t _e = (expr); if (_e != hipSuccess) { \
    fprintf(stderr, "HIP error %s at %s:%d\n", hipGetErrorString(_e), __FILE__, __LINE__); \
    return; } } while (0)

namespace {

#define LAS __attribute__((address_space(3)))
typedef int v2i __attribute__((ext_vector_type(2)));
typedef int v4i __attribute__((ext_vector_type(4)));
typedef float v4f __attribute__((ext_vector_type(4)));
typedef unsigned v4u __attribute__((ext_vector_type(4)));

constexpr int NR = 8;
constexpr int KTOT = 4096, NFULL = 16384, NLOC = 2048, MLOC = 512;
constexpr int RING_BYTES = 147456;
constexpr int CTL_OFF = RING_BYTES;
constexpr int CNT_OFF = CTL_OFF + 64;
constexpr int LDS_BYTES = CNT_OFF + 256;
constexpr int IMG_BYTES = 16384, RB_BYTES = 64 * IMG_BYTES;

constexpr size_t WS_TICKET = 0;
constexpr size_t WS_TMO = 2048;
constexpr size_t WS_FLAG = 4096;
constexpr size_t WS_DUMMY = 128u << 10;
constexpr size_t WS_X = 1u << 20;
constexpr size_t SHARD_BYTES = (size_t)MLOC * KTOT;
constexpr size_t WS_NEED = WS_X + 2 * NR * SHARD_BYTES;

constexpr unsigned long long DEADLINE = 50000000ull;
constexpr int NCHUNK = 64;
#define PUSH_W 2
constexpr int PW = PUSH_W;
#define SKEW_A 0
#define PF_BIG 2
#define EPI_DPP 1
#define POLL_SLEEP 0
#define WARM_DELAY 16
#define WARM_GAP 0
#define TLB_WARM 0
#define CW_WARM 1
constexpr unsigned TICKETS = 44;

struct Params {
    const char* x; const char* w; const float* sx; const float* sw; float* out;
    char* ws[NR];
    int rank;
};
#define STAMP(i) do {} while (0)

template <int V> struct IC { static constexpr int value = V; };
template <int N, int I = 0, class F> __device__ __forceinline__ void static_for(F&& f) { if constexpr (I < N) { f(IC<I>{}); static_for<N, I + 1>(f); } }
template <int N> __device__ __forceinline__ void waitv() { asm volatile("s_waitcnt vmcnt(%0)" :: "i"(N) : "memory"); }
__device__ __forceinline__ void bar() { asm volatile("" ::: "memory"); __builtin_amdgcn_s_barrier(); asm volatile("" ::: "memory"); }

__device__ __forceinline__ void lds_dma(unsigned voff, const char* base, unsigned ldst) {
    asm volatile("s_mov_b32 m0, %2\n\ts_nop 0\n\tglobal_load_lds_dwordx4 %0, %1" :: "v"(voff), "s"(base), "s"(__builtin_amdgcn_readfirstlane(ldst)) : "memory");
}

__device__ __forceinline__ const char* uniform_ptr(const char* q) {
    const unsigned long long v = (unsigned long long)q;
    const char* r = (const char*)(((unsigned long long)(unsigned)__builtin_amdgcn_readfirstlane((unsigned)(v >> 32)) << 32) |
                                  (unsigned)__builtin_amdgcn_readfirstlane((unsigned)v));
    asm volatile("s_nop 4" : "+s"(r));
    return r;
}

constexpr int SA = SKEW_A, SB_ = SA - 1, SBLK = SA ? 32 / SA : 0, SHEAD = SBLK * (SA + SB_);
__device__ __forceinline__ int order_rb(int i) { if constexpr (SA == 0) return i & 1; else return i < SHEAD ? (i % (SA + SB_)) & 1 : 1; }
__device__ __forceinline__ int order_kt(int i) {
    if constexpr (SA == 0) return i >> 1;
    else { if (i >= SHEAD) return SBLK * SB_ + (i - SHEAD);
           const int blk = i / (SA + SB_), r = i % (SA + SB_); return ((r & 1) ? SB_ : SA) * blk + (r >> 1); } }
__device__ __forceinline__ int order_pos(int rb, int kt) {
    if constexpr (SA == 0) return 2 * kt + rb;
    else { if (rb == 0) return (SA + SB_) * (kt / SA) + 2 * (kt % SA);
           return kt < SBLK * SB_ ? (SA + SB_) * (kt / SB_) + 2 * (kt % SB_) + 1 : SHEAD + kt - SBLK * SB_; } }

__device__ __forceinline__ float silu_dq(int a, float sk) {
    constexpr float L2E = 1.44269504088896341f;
    const float t = (float)a * sk;
    const float e = __builtin_amdgcn_exp2f(__builtin_amdgcn_fmed3f(t, -60.f * L2E, 60.f * L2E));
    return t * __builtin_amdgcn_rcpf(__builtin_fmaf(e, -L2E, -L2E));
}

#define OUT_ST 2
__device__ __forceinline__ void store_out(float* q, v4f o) {
    if (OUT_ST == 0) *(v4f*)q = o;
    else if (OUT_ST == 1) __builtin_nontemporal_store(o, (v4f*)q);
    else if (OUT_ST == 2) asm volatile("global_store_dwordx4 %0, %1, off sc1\n\ts_nop 1" :: "v"(q), "v"(o) : "memory");
    else if (OUT_ST == 3) asm volatile("global_store_dwordx4 %0, %1, off sc0 sc1\n\ts_nop 1" :: "v"(q), "v"(o) : "memory");
    else asm volatile("global_store_dwordx4 %0, %1, off sc1 nt\n\ts_nop 1" :: "v"(q), "v"(o) : "memory");
}

template <bool BIG> struct Cfg {
    static constexpr int MT = BIG ? 4 : 2, NT = BIG ? 4 : 2, WN = BIG ? 2 : 4, NSUB = BIG ? 1 : 2;
    static constexpr int NSTEP = BIG ? 64 : 32;
    static constexpr int RING = 6, PF = BIG ? PF_BIG : 4;
    static constexpr int A_BYTES = BIG ? 16384 : 8192, B_BYTES = BIG ? 8192 : 16384;
    static constexpr int A_OFF = BIG ? RING * 8192 : 0, B_OFF = BIG ? 0 : RING * 8192;
    static constexpr int L = 3;
    static constexpr int NBAR = 2 * NSTEP + 2;
    static_assert(RING * (A_BYTES + B_BYTES) <= RING_BYTES && PF <= RING - 1, "ring");
};

template <bool BIG, int SET>
__device__ __forceinline__ void gemm_tile_set(LAS char* lds, int wid, int lane, const char* ab, unsigned a_voff0, unsigned a_voff1, int a_step,
                                              const char* wb, float* out, float scale) {
    using C = Cfg<BIG>;
    constexpr int MT = C::MT, NT = C::NT, WN = C::WN, NSUB = C::NSUB, A_BYTES = C::A_BYTES, B_BYTES = C::B_BYTES, RING = C::RING, PF = C::PF, L = C::L;
    constexpr int NMAIN = C::NSTEP - PF, NLOOP = NMAIN / RING * RING;
    const unsigned lds0 = __builtin_amdgcn_readfirstlane((unsigned)(unsigned long long)lds);
    ab = uniform_ptr(ab); wb = uniform_ptr(wb);
    wid = __builtin_amdgcn_readfirstlane(wid); a_step = __builtin_amdgcn_readfirstlane(a_step);
    const int wm = wid / WN, wn = wid % WN;
    const int li = lane & 15, q = lane >> 4;
    auto wswz = [&](int k) { return (unsigned)(k * NFULL + (((lane & 7) ^ (((k >> 1) & 3) | (((k >> 4) & 1) << 2))) * 16)); };
    const unsigned b_voff0 = wswz((BIG ? wid * 8 : wid * 16) + (lane >> 3)), b_voff1 = wswz(wid * 16 + 8 + (lane >> 3));
    unsigned dA = lds0 + C::A_OFF + wid * (BIG ? 2048 : 1024), dB = lds0 + C::B_OFF + wid * (BIG ? 1024 : 2048);
    asm volatile("" : "+s"(dA), "+s"(dB));
    LAS char* ra0 = lds + C::A_OFF + (BIG ? (unsigned)((wm * MT * 16 + li) * 64 + ((q ^ ((-(li >> 2)) & 3)) * 16))
                                          : (unsigned)((wm * MT * 16 + li) * 128 + ((q ^ ((li >> 1) & 7)) * 16)));
    LAS char* ra1 = ra0 + (((q ^ ((li >> 1) & 7)) & 4) ? -64 : 64);
    LAS char* rb_[NT];
    const int fb = ((li >> 2) & 3) | ((q & 1) << 2);
#pragma unroll
    for (int n = 0; n < NT; ++n) rb_[n] = lds + C::B_OFF + (unsigned)((16 * q + (li >> 1)) * 128 + (((wn * NT + n) ^ fb) * 16) + 8 * (li & 1));
    v4i acc[MT][NT];
#pragma unroll
    for (int m = 0; m < MT; ++m)
#pragma unroll
        for (int n = 0; n < NT; ++n) acc[m][n] = (v4i){0, 0, 0, 0};

    auto dma = [&](auto ic, int sl) {
        constexpr int i = decltype(ic)::value;
        if constexpr (i == 0) lds_dma(a_voff0, ab, dA + (unsigned)(sl * A_BYTES));
        if constexpr (i == 1 && BIG) lds_dma(a_voff1, ab, dA + (unsigned)(sl * A_BYTES + 1024));
        if constexpr (i == 1 && !BIG) lds_dma(b_voff0, wb, dB + (unsigned)(sl * B_BYTES));
        if constexpr (i == 2 && BIG) lds_dma(b_voff0, wb, dB + (unsigned)(sl * B_BYTES));
        if constexpr (i == 2 && !BIG) lds_dma(b_voff1, wb, dB + (unsigned)(sl * B_BYTES + 1024));
        if constexpr (i == L - 1) { ab += a_step; wb += (BIG ? 64 : 128) * NFULL; }
    };
    auto issue = [&](int sl) { static_for<L>([&](auto ic) { dma(ic, sl); }); };
    auto step = [&](auto slotc, auto islotc, auto wvc) {
        constexpr int sl = decltype(slotc)::value, isl = decltype(islotc)::value, WV = decltype(wvc)::value;
        v4i wf[NSUB][NT], xf[NSUB][MT];
        auto rdw = [&](int s, int n) {
            const v2i lo = __builtin_amdgcn_ds_read_tr8_b64_v2i32((LAS v2i*)(rb_[n] + sl * B_BYTES + s * 8192));
            const v2i hi = __builtin_amdgcn_ds_read_tr8_b64_v2i32((LAS v2i*)(rb_[n] + sl * B_BYTES + s * 8192 + 1024));
            wf[s][n] = (v4i){lo.x, lo.y, hi.x, hi.y}; };
        auto rdx = [&](int s, int m) { xf[s][m] = *(LAS v4i*)((s ? ra1 : ra0) + sl * A_BYTES + m * (BIG ? 1024 : 2048)); };
        auto idma = [&](auto ic) { __builtin_amdgcn_sched_barrier(0); if constexpr (isl >= 0) dma(ic, isl); __builtin_amdgcn_sched_barrier(0); };
        if constexpr (BIG) {
            rdw(0, 0); rdw(0, 1); idma(IC<0>{});
            rdw(0, 2); rdw(0, 3); idma(IC<1>{});
            rdx(0, 0); rdx(0, 1); rdx(0, 2); rdx(0, 3); idma(IC<2>{});
        } else {
            rdw(0, 0); rdw(0, 1); idma(IC<0>{});
            rdx(0, 0); rdx(0, 1); rdw(1, 0); idma(IC<1>{});
            rdw(1, 1); rdx(1, 0); rdx(1, 1); idma(IC<2>{});
        }
        if constexpr (WV >= 0) waitv<WV * L>();
        asm volatile("s_waitcnt lgkmcnt(0)" ::: "memory");
        bar();
        __builtin_amdgcn_sched_barrier(0);
        __builtin_amdgcn_s_setprio(1);
#pragma unroll
        for (int s = 0; s < NSUB; ++s)
#pragma unroll
            for (int m = 0; m < MT; ++m)
#pragma unroll
                for (int n = 0; n < NT; ++n) acc[m][n] = __builtin_amdgcn_mfma_i32_16x16x64_i8(wf[s][n], xf[s][m], acc[m][n], 0, 0, 0);
        __builtin_amdgcn_s_setprio(0);
        __builtin_amdgcn_sched_barrier(0);
        bar();
    };
#pragma unroll
    for (int s = 0; s < PF; ++s) issue(s);
    waitv<(PF - 1) * L>();
    bar();
    if constexpr (SET == 1) bar();
    for (int j = 0; j < NLOOP; j += RING)
        static_for<RING>([&](auto ic) { constexpr int i = decltype(ic)::value; step(IC<i>{}, IC<(i + PF) % RING>{}, IC<PF - 1>{}); });
    static_for<NMAIN - NLOOP>([&](auto ic) { constexpr int i = decltype(ic)::value; step(IC<i % RING>{}, IC<(i + PF) % RING>{}, IC<PF - 1>{}); });
    static_for<PF>([&](auto ic) { constexpr int i = decltype(ic)::value;
        step(IC<(NMAIN + i) % RING>{}, IC<-1>{}, IC<PF - 2 - i>{}); });
    if constexpr (SET == 0) bar();
    const bool bad = __builtin_amdgcn_readfirstlane(((volatile LAS unsigned*)(lds + CTL_OFF))[1]) != 0u;
    float* op = out + (size_t)(wm * MT * 16 + (li & 7)) * NLOC + wn * NT * 16 + (li >> 3) * 16 + 4 * q;
    if (!bad) {
#pragma unroll
        for (int m = 0; m < MT; ++m)
#pragma unroll
            for (int n = 0; n < NT; n += 2) {
                v4i lo, hi;
#pragma unroll
                for (int e = 0; e < 4; ++e) {
                    lo[e] = __builtin_amdgcn_update_dpp(acc[m][n][e], acc[m][n + 1][e], 0x128, 0xf, 0xc, false);
                    hi[e] = __builtin_amdgcn_update_dpp(acc[m][n + 1][e], acc[m][n][e], 0x128, 0xf, 0x3, false);
                }
                v4f o; o.x = silu_dq(lo.x, scale); o.y = silu_dq(lo.y, scale); o.z = silu_dq(lo.z, scale); o.w = silu_dq(lo.w, scale);
                store_out(op + (size_t)m * 16 * NLOC + n * 16, o);
                o.x = silu_dq(hi.x, scale); o.y = silu_dq(hi.y, scale); o.z = silu_dq(hi.z, scale); o.w = silu_dq(hi.w, scale);
                store_out(op + (EPI_DPP ? (size_t)(m * 16 + 8) * NLOC + n * 16 : (size_t)m * 16 * NLOC + n * 16 + 16), o);
            }
    } else {
        const float qnan = __builtin_nanf("");
        for (int m = 0; m < MT; ++m)
            for (int n = 0; n < NT; n += 2) {
                store_out(op + (size_t)m * 16 * NLOC + n * 16, (v4f){qnan, qnan, qnan, qnan});
                store_out(op + (EPI_DPP ? (size_t)(m * 16 + 8) * NLOC + n * 16 : (size_t)m * 16 * NLOC + n * 16 + 16), (v4f){qnan, qnan, qnan, qnan});
            }
    }
}
template <bool BIG>
__device__ __forceinline__ void gemm_tile(LAS char* lds, int wid, int lane, const char* ab, unsigned a_voff0, unsigned a_voff1, int a_step,
                                          const char* wb, float* out, float scale) {
    wid = __builtin_amdgcn_readfirstlane(wid);
    if (wid < 4) gemm_tile_set<BIG, 0>(lds, wid, lane, ab, a_voff0, a_voff1, a_step, wb, out, scale);
    else gemm_tile_set<BIG, 1>(lds, wid, lane, ab, a_voff0, a_voff1, a_step, wb, out, scale);
}

__global__ __launch_bounds__(576) void fused_kernel(Params p) {
    extern __shared__ __attribute__((aligned(16))) char shm[];
    LAS char* lds = (LAS char*)shm;
    volatile LAS unsigned* ctl = (volatile LAS unsigned*)(lds + CTL_OFF);
    const int tid = threadIdx.x, lane = tid & 63, wid = __builtin_amdgcn_readfirstlane(tid >> 6);
    const int b = blockIdx.x, grp = b & 7, slot = b >> 3;
    char* myws = p.ws[p.rank];

    if (slot < 4) {
        if (wid >= 4) return;
        unsigned tkv; unsigned long long sv;
        asm volatile("s_mov_b64 %1, exec\n\ts_mov_b64 exec, 1\n\tglobal_atomic_add %0, %2, %3, %4 sc0\n\ts_mov_b64 exec, %1"
                     : "=&v"(tkv), "=&s"(sv) : "v"(0u), "v"(1u), "s"(myws + WS_TICKET + grp * 128) : "memory");
        if (grp == 7) { waitv<0>(); asm volatile("" :: "v"(tkv)); return; }
        ctl[16 + tid % 64] = 0u;
        asm volatile("s_waitcnt lgkmcnt(0)" ::: "memory");
        const int gw = slot * 4 + wid;
        const int dstr = (p.rank + 1 + grp) & 7;
        const char* xs = p.x;
        const int lr = lane >> 3, c8 = lane & 7;
        const unsigned soff = (unsigned)((16 * gw + lr) * KTOT + c8 * 16);
        unsigned doff[2];
#pragma unroll
        for (int i = 0; i < 2; ++i) { const int r = 16 * gw + 8 * i + lr;
            doff[i] = (unsigned)((c8 >> 2) * IMG_BYTES + r * 64 + (((c8 & 3) ^ ((-(r >> 2)) & 3)) * 16)); }
        v4u R[PW][2];
        auto ld = [&](v4u (&r)[2], int c) {
            const int cc = c < NCHUNK ? c : NCHUNK - 1;
            const char* sp = xs + (unsigned)(order_rb(cc) * 256 * KTOT + order_kt(cc) * 128);
            asm volatile("global_load_dwordx4 %0, %1, %2" : "=v"(r[0]) : "v"(soff), "s"(sp) : "memory");
            asm volatile("global_load_dwordx4 %0, %1, %2" : "=v"(r[1]) : "v"(soff + 8u * KTOT), "s"(sp) : "memory"); };
#pragma unroll
        for (int i = 0; i < PW; ++i) ld(R[i], i);
        unsigned tw0 = 0, tw1 = 0;
        unsigned tk;
        asm volatile("s_waitcnt vmcnt(%2)\n\tv_readfirstlane_b32 %0, %1" : "=s"(tk) : "v"(tkv), "i"(TLB_WARM ? 2 : 0) : "memory");
        bar();
        const unsigned call = tk / TICKETS + 1u;
        char* dst = p.ws[dstr] + WS_X + ((size_t)(call & 1u) * NR + p.rank) * SHARD_BYTES;
        char* flag = p.ws[dstr] + WS_FLAG + (size_t)p.rank * NCHUNK * 128 + slot * 4;
        const unsigned cnt0 = (unsigned)(unsigned long long)lds + CNT_OFF;
        auto publish = [&](int t) {
            unsigned old = 0;
            if (t >= 0) {
                if (lane == 0) asm volatile("ds_add_rtn_u32 %0, %1, %2\n\ts_waitcnt lgkmcnt(0)" : "=&v"(old) : "v"(cnt0 + (unsigned)t * 4u), "v"(1u) : "memory");
                old = __builtin_amdgcn_readfirstlane(old);
            }
            char* fp = (t >= 0 && old == 3u) ? flag + (size_t)t * 128 : myws + WS_DUMMY + (b * 4 + wid) * 128;
            if (lane == 0) asm volatile("global_store_dword %0, %1, off sc0 sc1" :: "v"(fp), "v"(call) : "memory"); };
        static_assert(NCHUNK % PW == 0, "unroll");
        for (int c0 = 0; c0 < NCHUNK; c0 += PW) static_for<PW>([&](auto ic) {
            constexpr int i = decltype(ic)::value;
            const int c = c0 + i;
            waitv<5 * (PW - 1)>();
            publish(c - PW);
            const char* dp = dst + (unsigned)(order_rb(c) * RB_BYTES + order_kt(c) * 2 * IMG_BYTES);
            asm volatile("" : "+v"(R[i][0]), "+v"(R[i][1]));
            asm volatile("global_store_dwordx4 %0, %1, %2 sc0 sc1\n\ts_nop 1" :: "v"(doff[0]), "v"(R[i][0]), "s"(dp) : "memory");
            asm volatile("global_store_dwordx4 %0, %1, %2 sc0 sc1\n\ts_nop 1" :: "v"(doff[1]), "v"(R[i][1]), "s"(dp) : "memory");
            ld(R[i], c + PW);
        });
        static_for<PW>([&](auto ic) {
            constexpr int v = decltype(ic)::value;
            waitv<2 + 5 * (PW - 1 - v) + v>();
            publish(NCHUNK - PW + v);
        });
        waitv<0>();
#pragma unroll
        for (int i = 0; i < PW; ++i) asm volatile("" :: "v"(R[i][0]), "v"(R[i][1]));
        asm volatile("" :: "v"(tw0), "v"(tw1));
        if (wid == 0) STAMP(7);
        return;
    }

    const int j = slot - 4, pj = j >> 2;
    const int h = grp >> 2;
    const int rb = (h ^ pj) & 1;
    const int src = (p.rank + 1 + pj) & 7;
    const int cb = (grp & 3) * 4 + (j & 3);
    const bool has_own = pj < 4;

    if (wid == 8) {
        unsigned tk = 0;
        if (lane == 0) tk = __hip_atomic_fetch_add((unsigned*)(myws + WS_TICKET + grp * 128), 1u, __ATOMIC_RELAXED, __HIP_MEMORY_SCOPE_AGENT);
        tk = __builtin_amdgcn_readfirstlane(tk);
        const unsigned call = tk / TICKETS + 1u;
        const char* flag = myws + WS_FLAG + (size_t)src * NCHUNK * 128 + (lane & 3) * 4;
        unsigned* tmo = (unsigned*)(myws + WS_TMO);
        int ready = 0;
        bool bad = false;
        auto poll = [&]() -> unsigned {
            int kt = ready + (lane >> 2); kt = kt < 32 ? kt : 31;
            const unsigned* q = lane < 60 ? (const unsigned*)(flag + order_pos(rb, kt) * 128) : tmo;
            const unsigned v = __hip_atomic_load(q, __ATOMIC_RELAXED, __HIP_MEMORY_SCOPE_SYSTEM);
            unsigned long long m = __ballot(v >= call);
            m = m & (m >> 1); m = m & (m >> 2);
            m = (~m & 0x1111111111111111ull) | (1ull << 60);
            ready += (int)(__builtin_ctzll(m) >> 2);
            ready = ready < 32 ? ready : 32;
            return __builtin_amdgcn_readlane(v, 60);
        };
        auto ensure = [&](int kt) {
            const int need = kt + 1;
            if (ready >= need) return;
            const unsigned long long t0 = __builtin_amdgcn_s_memrealtime();
            for (;;) {
                const unsigned tv = poll();
                if (ready >= need) {
                    break; }
                if (tv == call || __builtin_amdgcn_s_memrealtime() - t0 > DEADLINE) {
                    if (lane == 0) __hip_atomic_store(tmo, call, __ATOMIC_RELAXED, __HIP_MEMORY_SCOPE_AGENT);
                    bad = true; ready = 32; break;
                }
                if (POLL_SLEEP) __builtin_amdgcn_s_sleep(POLL_SLEEP);
            }
        };
        if (lane == 0) { ctl[0] = call; ctl[1] = 0u; }
        asm volatile("s_waitcnt lgkmcnt(0)" ::: "memory");
        unsigned wo;
        asm volatile("global_load_dword %0, %1, off" : "=v"(wo) : "v"(p.out + (size_t)(src * MLOC + rb * 256 + lane * 4) * NLOC + cb * 128) : "memory");
        if (has_own) for (int i = 0; i < Cfg<false>::NBAR; ++i) bar();
        waitv<0>();
        asm volatile("" :: "v"(wo));
        constexpr int NS = Cfg<true>::NSTEP, PFB = Cfg<true>::PF;
        if (has_own) poll();
        ensure(PFB >> 1);
        bar(); bar();
        for (int s = 0; s < NS; ++s) {
            bar();
            if (s + 1 + PFB < NS) ensure((s + 1 + PFB) >> 1);
            if (s == NS - 1) { if (lane == 0) ctl[1] = bad ? 1u : 0u; asm volatile("s_waitcnt lgkmcnt(0)" ::: "memory"); }
            bar();
        }
        bar();
        STAMP(6);
        return;
    }

    const float scale = (p.sx[0] * p.sw[0]) * -1.44269504088896341f;
    const char* wb = p.w + (size_t)p.rank * NLOC + cb * 128;
    if (has_own) {
        const int row0 = (h * 4 + pj) * 64, r = wid * 8 + (lane >> 3);
        gemm_tile<false>(lds, wid, lane, p.x + (size_t)row0 * KTOT, (unsigned)(r * KTOT + (((lane & 7) ^ ((r >> 1) & 7)) * 16)), 0u, 128,
                         wb, p.out + (size_t)(p.rank * MLOC + row0) * NLOC + cb * 128, scale);
        if (wid == 0) STAMP(0);
    }
    unsigned wr[5] = {0u, 0u, 0u, 0u, 0u};
    if (CW_WARM && !has_own) {
        if (WARM_DELAY) __builtin_amdgcn_s_sleep(WARM_DELAY);
        const int ww = (j - 16) * 8 + wid;
        const char* wq = p.w + (size_t)p.rank * NLOC + (grp & 3) * 512 + (size_t)(lane >> 2) * NFULL + (lane & 3) * 128;
        const char* xq = p.x + (size_t)h * 256 * KTOT + lane * 128;
#pragma unroll
        for (int i = 0; i < 3; ++i) {
            const int t = ww + 96 * i;
            asm volatile("global_load_dword %0, %1, off" : "=v"(wr[i]) : "v"(wq + (size_t)(t < 256 ? t : 255) * 16 * NFULL) : "memory");
            if (i < 2) asm volatile("global_load_dword %0, %1, off" : "=v"(wr[3 + i]) : "v"(xq + (size_t)(t < 128 ? t : 127) * 8192) : "memory");
            if (WARM_GAP) __builtin_amdgcn_s_sleep(WARM_GAP);
        }
    }
    bar();
    waitv<0>();
#pragma unroll
    for (int i = 0; i < 5; ++i) asm volatile("" :: "v"(wr[i]));
    const char* xsrc = p.ws[p.rank] + WS_X + ((size_t)(ctl[0] & 1u) * NR + src) * SHARD_BYTES + (size_t)rb * RB_BYTES;
    const unsigned lin0 = (unsigned)(wid * 2048 + lane * 16);
    gemm_tile<true>(lds, wid, lane, xsrc, lin0, lin0 + 1024, IMG_BYTES, wb, p.out + (size_t)(src * MLOC + rb * 256) * NLOC + cb * 128, scale);
}

}

extern "C" void kernel_launch(int rank, int nranks, ncclComm_t comm,
                              hipStream_t stream,
                              void* const* d_in, const int* in_sizes, int n_in,
                              void* d_out, int out_size,
                              const int* in_dtypes, int out_dtype,
                              void* const* d_out_all,
                              void* const* d_ws_all, size_t ws_bytes) {
    (void)comm; (void)in_sizes; (void)n_in; (void)out_size; (void)in_dtypes; (void)out_dtype; (void)d_out_all;
    if (nranks != NR || rank < 0 || rank >= NR || ws_bytes < WS_NEED) { fprintf(stderr, "unexpected configuration\n"); return; }
    HIP_CHECK(hipFuncSetAttribute((const void*)fused_kernel, hipFuncAttributeMaxDynamicSharedMemorySize, LDS_BYTES));
    Params p;
    p.x = (const char*)d_in[0]; p.w = (const char*)d_in[1]; p.sx = (const float*)d_in[2]; p.sw = (const float*)d_in[3];
    p.out = (float*)d_out; p.rank = rank;
    for (int i = 0; i < NR; ++i) p.ws[i] = (char*)d_ws_all[i];
    hipLaunchKernelGGL(fused_kernel, dim3(256), dim3(576), LDS_BYTES, stream, p);
    HIP_CHECK(hipGetLastError());
}
```

```cpp
#include <hip/hip_runtime.h>
#include <rccl/rccl.h>
#include <cstdio>

#define HIP_CHECK(expr) do { hipError_t _e = (expr); if (_e != hipSuccess) { \
    fprintf(stderr, "HIP error %s at %s:%d\n", hipGetErrorString(_e), __FILE__, __LINE__); \
    return; } } while (0)

namespace {

#define LAS __attribute__((address_space(3)))
typedef int v2i __attribute__((ext_vector_type(2)));
typedef int v4i __attribute__((ext_vector_type(4)));
typedef float v4f __attribute__((ext_vector_type(4)));
typedef unsigned v4u __attribute__((ext_vector_type(4)));

constexpr int NR = 8;
constexpr int KTOT = 4096, NFULL = 16384, NLOC = 2048, MLOC = 512;
constexpr int RING_BYTES = 147456;
constexpr int CTL_OFF = RING_BYTES;
constexpr int CNT_OFF = CTL_OFF + 64;
constexpr int LDS_BYTES = CNT_OFF + 256;
constexpr int IMG_BYTES = 16384, RB_BYTES = 64 * IMG_BYTES;

constexpr size_t WS_TICKET = 0;
constexpr size_t WS_TMO = 2048;
constexpr size_t WS_FLAG = 4096;
constexpr size_t WS_DUMMY = 128u << 10;
constexpr size_t WS_X = 1u << 20;
constexpr size_t SHARD_BYTES = (size_t)MLOC * KTOT;
constexpr size_t WS_NEED = WS_X + 2 * NR * SHARD_BYTES;

constexpr unsigned long long DEADLINE = 50000000ull;
constexpr int NCHUNK = 64;
constexpr int PW = 2;
constexpr int PF_BIG = 4, PF_END = 2;
constexpr unsigned TICKETS = 44;

struct Params {
    const char* x; const char* w; const float* sx; const float* sw; float* out;
    char* ws[NR];
    int rank;
};
#define STAMP(i) do {} while (0)

template <int V> struct IC { static constexpr int value = V; };
template <int N, int I = 0, class F> __device__ __forceinline__ void static_for(F&& f) { if constexpr (I < N) { f(IC<I>{}); static_for<N, I + 1>(f); } }
template <int N> __device__ __forceinline__ void waitv() { asm volatile("s_waitcnt vmcnt(%0)" :: "i"(N) : "memory"); }
__device__ __forceinline__ void bar() { asm volatile("" ::: "memory"); __builtin_amdgcn_s_barrier(); asm volatile("" ::: "memory"); }

__device__ __forceinline__ void lds_dma(unsigned voff, const char* base, unsigned ldst) {
    asm volatile("s_mov_b32 m0, %2\n\ts_nop 0\n\tglobal_load_lds_dwordx4 %0, %1" :: "v"(voff), "s"(base), "s"(__builtin_amdgcn_readfirstlane(ldst)) : "memory");
}

__device__ __forceinline__ const char* uniform_ptr(const char* q) {
    const unsigned long long v = (unsigned long long)q;
    const char* r = (const char*)(((unsigned long long)(unsigned)__builtin_amdgcn_readfirstlane((unsigned)(v >> 32)) << 32) |
                                  (unsigned)__builtin_amdgcn_readfirstlane((unsigned)v));
    asm volatile("s_nop 4" : "+s"(r));
    return r;
}

__device__ __forceinline__ int order_rb(int c) { return c & 1; }
__device__ __forceinline__ int order_kt(int c) { return c >> 1; }
__device__ __forceinline__ int order_pos(int rb, int kt) { return 2 * kt + rb; }

__device__ __forceinline__ float silu_dq(int a, float sk) {
    constexpr float L2E = 1.44269504088896341f;
    const float t = (float)a * sk;
    const float e = __builtin_amdgcn_exp2f(__builtin_amdgcn_fmed3f(t, -60.f * L2E, 60.f * L2E));
    return t * __builtin_amdgcn_rcpf(__builtin_fmaf(e, -L2E, -L2E));
}

__device__ __forceinline__ void store_out(float* q, v4f o) {
    asm volatile("global_store_dwordx4 %0, %1, off sc1\n\ts_nop 1" :: "v"(q), "v"(o) : "memory");
}

template <bool BIG> struct Cfg {
    static constexpr int MT = BIG ? 4 : 2, NT = BIG ? 4 : 2, WN = BIG ? 2 : 4, NSUB = BIG ? 1 : 2;
    static constexpr int NSTEP = BIG ? 64 : 32;
    static constexpr int RING = 6, PF = BIG ? PF_BIG : 4, PFE = BIG ? PF_END : 4;
    static constexpr int A_BYTES = BIG ? 16384 : 8192, B_BYTES = BIG ? 8192 : 16384;
    static constexpr int A_OFF = BIG ? RING * 8192 : 0, B_OFF = BIG ? 0 : RING * 8192;
    static constexpr int L = 3;
    static constexpr int NBAR = 2 * NSTEP + 2;
    static_assert(RING * (A_BYTES + B_BYTES) <= RING_BYTES && PF <= RING - 1 && PFE <= PF, "ring");
    static constexpr int NLATE = PFE < PF ? 4 : 0, JSW = NSTEP - NLATE - PF;
    __host__ __device__ static constexpr int stage_at(int j) { return j < JSW ? j + PF : (j + PFE >= NSTEP - NLATE && j + PFE < NSTEP ? j + PFE : -1); }
    __host__ __device__ static constexpr int issued_upto(int j) { int u = PF - 1; for (int i = 0; i <= j; ++i) if (stage_at(i) >= 0) u = stage_at(i); return u; }
};

template <bool BIG, int SET>
__device__ __forceinline__ void gemm_tile_set(LAS char* lds, int wid, int lane, const char* ab, unsigned a_voff0, unsigned a_voff1, int a_step,
                                              const char* wb, float* out, float scale) {
    using C = Cfg<BIG>;
    constexpr int MT = C::MT, NT = C::NT, WN = C::WN, NSUB = C::NSUB, A_BYTES = C::A_BYTES, B_BYTES = C::B_BYTES, RING = C::RING, PF = C::PF, L = C::L;
    constexpr int NLOOP = C::JSW / RING * RING;
    const unsigned lds0 = __builtin_amdgcn_readfirstlane((unsigned)(unsigned long long)lds);
    ab = uniform_ptr(ab); wb = uniform_ptr(wb);
    wid = __builtin_amdgcn_readfirstlane(wid); a_step = __builtin_amdgcn_readfirstlane(a_step);
    const int wm = wid / WN, wn = wid % WN;
    const int li = lane & 15, q = lane >> 4;
    auto wswz = [&](int k) { return (unsigned)(k * NFULL + (((lane & 7) ^ (((k >> 1) & 3) | (((k >> 4) & 1) << 2))) * 16)); };
    const unsigned b_voff0 = wswz((BIG ? wid * 8 : wid * 16) + (lane >> 3)), b_voff1 = wswz(wid * 16 + 8 + (lane >> 3));
    unsigned dA = lds0 + C::A_OFF + wid * (BIG ? 2048 : 1024), dB = lds0 + C::B_OFF + wid * (BIG ? 1024 : 2048);
    asm volatile("" : "+s"(dA), "+s"(dB));
    LAS char* ra0 = lds + C::A_OFF + (BIG ? (unsigned)((wm * MT * 16 + li) * 64 + ((q ^ ((-(li >> 2)) & 3)) * 16))
                                          : (unsigned)((wm * MT * 16 + li) * 128 + ((q ^ ((li >> 1) & 7)) * 16)));
    LAS char* ra1 = ra0 + (((q ^ ((li >> 1) & 7)) & 4) ? -64 : 64);
    LAS char* rb_[NT];
    const int fb = ((li >> 2) & 3) | ((q & 1) << 2);
#pragma unroll
    for (int n = 0; n < NT; ++n) rb_[n] = lds + C::B_OFF + (unsigned)((16 * q + (li >> 1)) * 128 + (((wn * NT + n) ^ fb) * 16) + 8 * (li & 1));
    v4i acc[MT][NT];
#pragma unroll
    for (int m = 0; m < MT; ++m)
#pragma unroll
        for (int n = 0; n < NT; ++n) acc[m][n] = (v4i){0, 0, 0, 0};

    auto dma = [&](auto ic, int sl) {
        constexpr int i = decltype(ic)::value;
        if constexpr (i == 0) lds_dma(a_voff0, ab, dA + (unsigned)(sl * A_BYTES));
        if constexpr (i == 1 && BIG) lds_dma(a_voff1, ab, dA + (unsigned)(sl * A_BYTES + 1024));
        if constexpr (i == 1 && !BIG) lds_dma(b_voff0, wb, dB + (unsigned)(sl * B_BYTES));
        if constexpr (i == 2 && BIG) lds_dma(b_voff0, wb, dB + (unsigned)(sl * B_BYTES));
        if constexpr (i == 2 && !BIG) lds_dma(b_voff1, wb, dB + (unsigned)(sl * B_BYTES + 1024));
        if constexpr (i == L - 1) { ab += a_step; wb += (BIG ? 64 : 128) * NFULL; }
    };
    auto issue = [&](int sl) { static_for<L>([&](auto ic) { dma(ic, sl); }); };
    auto step = [&](auto slotc, auto islotc, auto wvc) {
        constexpr int sl = decltype(slotc)::value, isl = decltype(islotc)::value, WV = decltype(wvc)::value;
        v4i wf[NSUB][NT], xf[NSUB][MT];
        auto rdw = [&](int s, int n) {
            const v2i lo = __builtin_amdgcn_ds_read_tr8_b64_v2i32((LAS v2i*)(rb_[n] + sl * B_BYTES + s * 8192));
            const v2i hi = __builtin_amdgcn_ds_read_tr8_b64_v2i32((LAS v2i*)(rb_[n] + sl * B_BYTES + s * 8192 + 1024));
            wf[s][n] = (v4i){lo.x, lo.y, hi.x, hi.y}; };
        auto rdx = [&](int s, int m) { xf[s][m] = *(LAS v4i*)((s ? ra1 : ra0) + sl * A_BYTES + m * (BIG ? 1024 : 2048)); };
        auto idma = [&](auto ic) { __builtin_amdgcn_sched_barrier(0); if constexpr (isl >= 0) dma(ic, isl); __builtin_amdgcn_sched_barrier(0); };
        if constexpr (BIG) {
            rdw(0, 0); rdw(0, 1); idma(IC<0>{});
            rdw(0, 2); rdw(0, 3); idma(IC<1>{});
            rdx(0, 0); rdx(0, 1); rdx(0, 2); rdx(0, 3); idma(IC<2>{});
        } else {
            rdw(0, 0); rdw(0, 1); idma(IC<0>{});
            rdx(0, 0); rdx(0, 1); rdw(1, 0); idma(IC<1>{});
            rdw(1, 1); rdx(1, 0); rdx(1, 1); idma(IC<2>{});
        }
        if constexpr (WV >= 0) waitv<WV * L>();
        asm volatile("s_waitcnt lgkmcnt(0)" ::: "memory");
        bar();
        __builtin_amdgcn_sched_barrier(0);
        __builtin_amdgcn_s_setprio(1);
#pragma unroll
        for (int s = 0; s < NSUB; ++s)
#pragma unroll
            for (int m = 0; m < MT; ++m)
#pragma unroll
                for (int n = 0; n < NT; ++n) acc[m][n] = __builtin_amdgcn_mfma_i32_16x16x64_i8(wf[s][n], xf[s][m], acc[m][n], 0, 0, 0);
        __builtin_amdgcn_s_setprio(0);
        __builtin_amdgcn_sched_barrier(0);
        bar();
    };
#pragma unroll
    for (int s = 0; s < PF; ++s) issue(s);
    waitv<(PF - 1) * L>();
    bar();
    if constexpr (SET == 1) bar();
    for (int j = 0; j < NLOOP; j += RING)
        static_for<RING>([&](auto ic) { constexpr int i = decltype(ic)::value; step(IC<i>{}, IC<(i + PF) % RING>{}, IC<PF - 1>{}); });
    static_for<C::NSTEP - NLOOP>([&](auto ic) {
        constexpr int j = NLOOP + decltype(ic)::value, st = C::stage_at(j);
        step(IC<j % RING>{}, IC<(st >= 0 ? st % RING : -1)>{}, IC<(j + 1 < C::NSTEP ? C::issued_upto(j) - (j + 1) : -1)>{}); });
    if constexpr (SET == 0) bar();
    const bool bad = __builtin_amdgcn_readfirstlane(((volatile LAS unsigned*)(lds + CTL_OFF))[1]) != 0u;
    float* op = out + (size_t)(wm * MT * 16 + (li & 7)) * NLOC + wn * NT * 16 + (li >> 3) * 16 + 4 * q;
    if (!bad) {
#pragma unroll
        for (int m = 0; m < MT; ++m)
#pragma unroll
            for (int n = 0; n < NT; n += 2) {
                v4i lo, hi;
#pragma unroll
                for (int e = 0; e < 4; ++e) {
                    lo[e] = __builtin_amdgcn_update_dpp(acc[m][n][e], acc[m][n + 1][e], 0x128, 0xf, 0xc, false);
                    hi[e] = __builtin_amdgcn_update_dpp(acc[m][n + 1][e], acc[m][n][e], 0x128, 0xf, 0x3, false);
                }
                v4f o; o.x = silu_dq(lo.x, scale); o.y = silu_dq(lo.y, scale); o.z = silu_dq(lo.z, scale); o.w = silu_dq(lo.w, scale);
                store_out(op + (size_t)m * 16 * NLOC + n * 16, o);
                o.x = silu_dq(hi.x, scale); o.y = silu_dq(hi.y, scale); o.z = silu_dq(hi.z, scale); o.w = silu_dq(hi.w, scale);
                store_out(op + (size_t)(m * 16 + 8) * NLOC + n * 16, o);
            }
    } else {
        const float qnan = __builtin_nanf("");
        for (int m = 0; m < MT; ++m)
            for (int n = 0; n < NT; n += 2) {
                store_out(op + (size_t)m * 16 * NLOC + n * 16, (v4f){qnan, qnan, qnan, qnan});
                store_out(op + (size_t)(m * 16 + 8) * NLOC + n * 16, (v4f){qnan, qnan, qnan, qnan});
            }
    }
}
template <bool BIG>
__device__ __forceinline__ void gemm_tile(LAS char* lds, int wid, int lane, const char* ab, unsigned a_voff0, unsigned a_voff1, int a_step,
                                          const char* wb, float* out, float scale) {
    wid = __builtin_amdgcn_readfirstlane(wid);
    if (wid < 4) gemm_tile_set<BIG, 0>(lds, wid, lane, ab, a_voff0, a_voff1, a_step, wb, out, scale);
    else gemm_tile_set<BIG, 1>(lds, wid, lane, ab, a_voff0, a_voff1, a_step, wb, out, scale);
}

__global__ __launch_bounds__(576) void fused_kernel(Params p) {
    extern __shared__ __attribute__((aligned(16))) char shm[];
    LAS char* lds = (LAS char*)shm;
    volatile LAS unsigned* ctl = (volatile LAS unsigned*)(lds + CTL_OFF);
    const int tid = threadIdx.x, lane = tid & 63, wid = __builtin_amdgcn_readfirstlane(tid >> 6);
    const int b = blockIdx.x, grp = b & 7, slot = b >> 3;
    char* myws = p.ws[p.rank];

    if (slot < 4) {
        if (wid >= 4) return;
        unsigned tkv; unsigned long long sv;
        asm volatile("s_mov_b64 %1, exec\n\ts_mov_b64 exec, 1\n\tglobal_atomic_add %0, %2, %3, %4 sc0\n\ts_mov_b64 exec, %1"
                     : "=&v"(tkv), "=&s"(sv) : "v"(0u), "v"(1u), "s"(myws + WS_TICKET + grp * 128) : "memory");
        if (grp == 7) { waitv<0>(); asm volatile("" :: "v"(tkv)); return; }
        ctl[16 + tid % 64] = 0u;
        asm volatile("s_waitcnt lgkmcnt(0)" ::: "memory");
        const int gw = slot * 4 + wid;
        const int dstr = (p.rank + 1 + grp) & 7;
        const char* xs = p.x;
        const int lr = lane >> 3, c8 = lane & 7;
        const unsigned soff = (unsigned)((16 * gw + lr) * KTOT + c8 * 16);
        unsigned doff[2];
#pragma unroll
        for (int i = 0; i < 2; ++i) { const int r = 16 * gw + 8 * i + lr;
            doff[i] = (unsigned)((c8 >> 2) * IMG_BYTES + r * 64 + (((c8 & 3) ^ ((-(r >> 2)) & 3)) * 16)); }
        v4u R[PW][2];
        auto ld = [&](v4u (&r)[2], int c) {
            const int cc = c < NCHUNK ? c : NCHUNK - 1;
            const char* sp = xs + (unsigned)(order_rb(cc) * 256 * KTOT + order_kt(cc) * 128);
            asm volatile("global_load_dwordx4 %0, %1, %2" : "=v"(r[0]) : "v"(soff), "s"(sp) : "memory");
            asm volatile("global_load_dwordx4 %0, %1, %2" : "=v"(r[1]) : "v"(soff + 8u * KTOT), "s"(sp) : "memory"); };
#pragma unroll
        for (int i = 0; i < PW; ++i) ld(R[i], i);
        unsigned tk;
        asm volatile("s_waitcnt vmcnt(0)\n\tv_readfirstlane_b32 %0, %1" : "=s"(tk) : "v"(tkv) : "memory");
        bar();
        const unsigned call = tk / TICKETS + 1u;
        char* dst = p.ws[dstr] + WS_X + ((size_t)(call & 1u) * NR + p.rank) * SHARD_BYTES;
        char* flag = p.ws[dstr] + WS_FLAG + (size_t)p.rank * NCHUNK * 128 + slot * 4;
        const unsigned cnt0 = (unsigned)(unsigned long long)lds + CNT_OFF;
        auto publish = [&](int t) {
            unsigned old = 0;
            if (t >= 0) {
                if (lane == 0) asm volatile("ds_add_rtn_u32 %0, %1, %2\n\ts_waitcnt lgkmcnt(0)" : "=&v"(old) : "v"(cnt0 + (unsigned)t * 4u), "v"(1u) : "memory");
                old = __builtin_amdgcn_readfirstlane(old);
            }
            char* fp = (t >= 0 && old == 3u) ? flag + (size_t)t * 128 : myws + WS_DUMMY + (b * 4 + wid) * 128;
            if (lane == 0) asm volatile("global_store_dword %0, %1, off sc0 sc1" :: "v"(fp), "v"(call) : "memory"); };
        static_assert(NCHUNK % PW == 0, "unroll");
        for (int c0 = 0; c0 < NCHUNK; c0 += PW) static_for<PW>([&](auto ic) {
            constexpr int i = decltype(ic)::value;
            const int c = c0 + i;
            waitv<5 * (PW - 1)>();
            publish(c - PW);
            const char* dp = dst + (unsigned)(order_rb(c) * RB_BYTES + order_kt(c) * 2 * IMG_BYTES);
            asm volatile("" : "+v"(R[i][0]), "+v"(R[i][1]));
            asm volatile("global_store_dwordx4 %0, %1, %2 sc0 sc1\n\ts_nop 1" :: "v"(doff[0]), "v"(R[i][0]), "s"(dp) : "memory");
            asm volatile("global_store_dwordx4 %0, %1, %2 sc0 sc1\n\ts_nop 1" :: "v"(doff[1]), "v"(R[i][1]), "s"(dp) : "memory");
            ld(R[i], c + PW);
        });
        static_for<PW>([&](auto ic) {
            constexpr int v = decltype(ic)::value;
            waitv<2 + 5 * (PW - 1 - v) + v>();
            publish(NCHUNK - PW + v);
        });
        waitv<0>();
#pragma unroll
        for (int i = 0; i < PW; ++i) asm volatile("" :: "v"(R[i][0]), "v"(R[i][1]));
        if (wid == 0) STAMP(7);
        return;
    }

    const int j = slot - 4, pj = j >> 2;
    const int h = grp >> 2;
    const int rb = (h ^ pj) & 1;
    const int src = (p.rank + 1 + pj) & 7;
    const int cb = (grp & 3) * 4 + (j & 3);
    const bool has_own = pj < 4;

    if (wid == 8) {
        unsigned tk = 0;
        if (lane == 0) tk = __hip_atomic_fetch_add((unsigned*)(myws + WS_TICKET + grp * 128), 1u, __ATOMIC_RELAXED, __HIP_MEMORY_SCOPE_AGENT);
        tk = __builtin_amdgcn_readfirstlane(tk);
        const unsigned call = tk / TICKETS + 1u;
        const char* flag = myws + WS_FLAG + (size_t)src * NCHUNK * 128 + (lane & 3) * 4;
        unsigned* tmo = (unsigned*)(myws + WS_TMO);
        int ready = 0;
        bool bad = false;
        auto poll = [&]() -> unsigned {
            int kt = ready + (lane >> 2); kt = kt < 32 ? kt : 31;
            const unsigned* q = lane < 60 ? (const unsigned*)(flag + order_pos(rb, kt) * 128) : tmo;
            const unsigned v = __hip_atomic_load(q, __ATOMIC_RELAXED, __HIP_MEMORY_SCOPE_SYSTEM);
            unsigned long long m = __ballot(v >= call);
            m = m & (m >> 1); m = m & (m >> 2);
            m = (~m & 0x1111111111111111ull) | (1ull << 60);
            ready += (int)(__builtin_ctzll(m) >> 2);
            ready = ready < 32 ? ready : 32;
            return __builtin_amdgcn_readlane(v, 60);
        };
        auto ensure = [&](int kt) {
            const int need = kt + 1;
            if (ready >= need) return;
            const unsigned long long t0 = __builtin_amdgcn_s_memrealtime();
            for (;;) {
                const unsigned tv = poll();
                if (ready >= need) {
                    break; }
                if (tv == call || __builtin_amdgcn_s_memrealtime() - t0 > DEADLINE) {
                    if (lane == 0) __hip_atomic_store(tmo, call, __ATOMIC_RELAXED, __HIP_MEMORY_SCOPE_AGENT);
                    bad = true; ready = 32; break;
                }
            }
        };
        if (lane == 0) { ctl[0] = call; ctl[1] = 0u; }
        asm volatile("s_waitcnt lgkmcnt(0)" ::: "memory");
        unsigned wo;
        asm volatile("global_load_dword %0, %1, off" : "=v"(wo) : "v"(p.out + (size_t)(src * MLOC + rb * 256 + lane * 4) * NLOC + cb * 128) : "memory");
        if (has_own) for (int i = 0; i < Cfg<false>::NBAR; ++i) bar();
        waitv<0>();
        asm volatile("" :: "v"(wo));
        using CB = Cfg<true>;
        if (has_own) poll();
        ensure(CB::PF >> 1);
        __builtin_amdgcn_fence(__ATOMIC_ACQUIRE, "");
        waitv<0>();
        bar(); bar();
        for (int s = 0; s < CB::NSTEP; ++s) {
            bar();
            const int st = CB::stage_at(s + 1);
            if (st >= 0) ensure(st >> 1);
            if (s == CB::NSTEP - 1) { if (lane == 0) ctl[1] = bad ? 1u : 0u; asm volatile("s_waitcnt lgkmcnt(0)" ::: "memory"); }
            bar();
        }
        bar();
        STAMP(6);
        return;
    }

    const float scale = (p.sx[0] * p.sw[0]) * -1.44269504088896341f;
    const char* wb = p.w + (size_t)p.rank * NLOC + cb * 128;
    if (has_own) {
        const int row0 = (h * 4 + pj) * 64, r = wid * 8 + (lane >> 3);
        gemm_tile<false>(lds, wid, lane, p.x + (size_t)row0 * KTOT, (unsigned)(r * KTOT + (((lane & 7) ^ ((r >> 1) & 7)) * 16)), 0u, 128,
                         wb, p.out + (size_t)(p.rank * MLOC + row0) * NLOC + cb * 128, scale);
        if (wid == 0) STAMP(0);
    }
    unsigned wr[5] = {0u, 0u, 0u, 0u, 0u};
    if (!has_own) {
        __builtin_amdgcn_s_sleep(16);
        const int ww = (j - 16) * 8 + wid;
        const char* wq = p.w + (size_t)p.rank * NLOC + (grp & 3) * 512 + (size_t)(lane >> 2) * NFULL + (lane & 3) * 128;
        const char* xq = p.x + (size_t)h * 256 * KTOT + lane * 128;
#pragma unroll
        for (int i = 0; i < 3; ++i) {
            const int t = ww + 96 * i;
            asm volatile("global_load_dword %0, %1, off" : "=v"(wr[i]) : "v"(wq + (size_t)(t < 256 ? t : 255) * 16 * NFULL) : "memory");
            if (i < 2) asm volatile("global_load_dword %0, %1, off" : "=v"(wr[3 + i]) : "v"(xq + (size_t)(t < 128 ? t : 127) * 8192) : "memory");
        }
    }
    bar();
    waitv<0>();
#pragma unroll
    for (int i = 0; i < 5; ++i) asm volatile("" :: "v"(wr[i]));
    const char* xsrc = p.ws[p.rank] + WS_X + ((size_t)(ctl[0] & 1u) * NR + src) * SHARD_BYTES + (size_t)rb * RB_BYTES;
    const unsigned lin0 = (unsigned)(wid * 2048 + lane * 16);
    gemm_tile<true>(lds, wid, lane, xsrc, lin0, lin0 + 1024, IMG_BYTES, wb, p.out + (size_t)(src * MLOC + rb * 256) * NLOC + cb * 128, scale);
}

}

extern "C" void kernel_launch(int rank, int nranks, ncclComm_t comm,
                              hipStream_t stream,
                              void* const* d_in, const int* in_sizes, int n_in,
                              void* d_out, int out_size,
                              const int* in_dtypes, int out_dtype,
                              void* const* d_out_all,
                              void* const* d_ws_all, size_t ws_bytes) {
    (void)comm; (void)in_sizes; (void)n_in; (void)out_size; (void)in_dtypes; (void)out_dtype; (void)d_out_all;
    if (nranks != NR || rank < 0 || rank >= NR || ws_bytes < WS_NEED) { fprintf(stderr, "unexpected configuration\n"); return; }
    HIP_CHECK(hipFuncSetAttribute((const void*)fused_kernel, hipFuncAttributeMaxDynamicSharedMemorySize, LDS_BYTES));
    Params p;
    p.x = (const char*)d_in[0]; p.w = (const char*)d_in[1]; p.sx = (const float*)d_in[2]; p.sw = (const float*)d_in[3];
    p.out = (float*)d_out; p.rank = rank;
    for (int i = 0; i < NR; ++i) p.ws[i] = (char*)d_ws_all[i];
    hipLaunchKernelGGL(fused_kernel, dim3(256), dim3(576), LDS_BYTES, stream, p);
    HIP_CHECK(hipGetLastError());
}
```

```cpp
#include <hip/hip_runtime.h>
#include <rccl/rccl.h>
#include <cstdio>

#define HIP_CHECK(expr) do { hipError_t _e = (expr); if (_e != hipSuccess) { \
    fprintf(stderr, "HIP error %s at %s:%d\n", hipGetErrorString(_e), __FILE__, __LINE__); \
    return; } } while (0)

namespace {

#define LAS __attribute__((address_space(3)))
typedef int v2i __attribute__((ext_vector_type(2)));
typedef int v4i __attribute__((ext_vector_type(4)));
typedef float v4f __attribute__((ext_vector_type(4)));
typedef unsigned v4u __attribute__((ext_vector_type(4)));

constexpr int NR = 8;
constexpr int KTOT = 4096, NFULL = 16384, NLOC = 2048, MLOC = 512;
constexpr int RING_BYTES = 147456;
constexpr int CTL_OFF = RING_BYTES;
constexpr int CNT_OFF = CTL_OFF + 64;
constexpr int LDS_BYTES = CNT_OFF + 256;
constexpr int IMG_BYTES = 16384, RB_BYTES = 64 * IMG_BYTES;

constexpr size_t WS_TICKET = 0;
constexpr size_t WS_TMO = 2048;
constexpr size_t WS_FLAG = 4096;
constexpr size_t WS_DUMMY = 128u << 10;
constexpr size_t WS_X = 1u << 20;
constexpr size_t SHARD_BYTES = (size_t)MLOC * KTOT;
constexpr size_t WS_NEED = WS_X + 2 * NR * SHARD_BYTES;

constexpr unsigned long long DEADLINE = 50000000ull;
constexpr int NCHUNK = 64;
constexpr int PW = 2;
constexpr int PF_BIG = 2, PF_END = 2;
constexpr unsigned TICKETS = 44;

struct Params {
    const char* x; const char* w; const float* sx; const float* sw; float* out;
    char* ws[NR];
    int rank;
};
#define STAMP(i) do {} while (0)

template <int V> struct IC { static constexpr int value = V; };
template <int N, int I = 0, class F> __device__ __forceinline__ void static_for(F&& f) { if constexpr (I < N) { f(IC<I>{}); static_for<N, I + 1>(f); } }
template <int N> __device__ __forceinline__ void waitv() { asm volatile("s_waitcnt vmcnt(%0)" :: "i"(N) : "memory"); }
__device__ __forceinline__ void bar() { asm volatile("" ::: "memory"); __builtin_amdgcn_s_barrier(); asm volatile("" ::: "memory"); }

__device__ __forceinline__ void lds_dma(unsigned voff, const char* base, unsigned ldst) {
    asm volatile("s_mov_b32 m0, %2\n\ts_nop 0\n\tglobal_load_lds_dwordx4 %0, %1" :: "v"(voff), "s"(base), "s"(__builtin_amdgcn_readfirstlane(ldst)) : "memory");
}

__device__ __forceinline__ const char* uniform_ptr(const char* q) {
    const unsigned long long v = (unsigned long long)q;
    const char* r = (const char*)(((unsigned long long)(unsigned)__builtin_amdgcn_readfirstlane((unsigned)(v >> 32)) << 32) |
                                  (unsigned)__builtin_amdgcn_readfirstlane((unsigned)v));
    asm volatile("s_nop 4" : "+s"(r));
    return r;
}

__device__ __forceinline__ int order_rb(int c) { return c & 1; }
__device__ __forceinline__ int order_kt(int c) { return c >> 1; }
__device__ __forceinline__ int order_pos(int rb, int kt) { return 2 * kt + rb; }

__device__ __forceinline__ float silu_dq(int a, float sk) {
    constexpr float L2E = 1.44269504088896341f;
    const float t = (float)a * sk;
    const float e = __builtin_amdgcn_exp2f(__builtin_amdgcn_fmed3f(t, -60.f * L2E, 60.f * L2E));
    return t * __builtin_amdgcn_rcpf(__builtin_fmaf(e, -L2E, -L2E));
}

__device__ __forceinline__ void store_out(float* q, v4f o) {
    asm volatile("global_store_dwordx4 %0, %1, off sc1\n\ts_nop 1" :: "v"(q), "v"(o) : "memory");
}

template <bool BIG> struct Cfg {
    static constexpr int MT = BIG ? 4 : 2, NT = BIG ? 4 : 2, WN = BIG ? 2 : 4, NSUB = BIG ? 1 : 2;
    static constexpr int NSTEP = BIG ? 64 : 32;
    static constexpr int RING = 6, PF = BIG ? PF_BIG : 4, PFE = BIG ? PF_END : 4;
    static constexpr int A_BYTES = BIG ? 16384 : 8192, B_BYTES = BIG ? 8192 : 16384;
    static constexpr int A_OFF = BIG ? RING * 8192 : 0, B_OFF = BIG ? 0 : RING * 8192;
    static constexpr int L = 3;
    static constexpr int NBAR = 2 * NSTEP + 2;
    static_assert(RING * (A_BYTES + B_BYTES) <= RING_BYTES && PF <= RING - 1 && PFE <= PF, "ring");
    static constexpr int NLATE = PFE < PF ? 4 : 0, JSW = NSTEP - NLATE - PF;
    __host__ __device__ static constexpr int stage_at(int j) { return j < JSW ? j + PF : (j + PFE >= NSTEP - NLATE && j + PFE < NSTEP ? j + PFE : -1); }
    __host__ __device__ static constexpr int issued_upto(int j) { int u = PF - 1; for (int i = 0; i <= j; ++i) if (stage_at(i) >= 0) u = stage_at(i); return u; }
};

template <bool BIG, int SET>
__device__ __forceinline__ void gemm_tile_set(LAS char* lds, int wid, int lane, const char* ab, unsigned a_voff0, unsigned a_voff1, int a_step,
                                              const char* wb, float* out, float scale) {
    using C = Cfg<BIG>;
    constexpr int MT = C::MT, NT = C::NT, WN = C::WN, NSUB = C::NSUB, A_BYTES = C::A_BYTES, B_BYTES = C::B_BYTES, RING = C::RING, PF = C::PF, L = C::L;
    constexpr int NLOOP = C::JSW / RING * RING;
    const unsigned lds0 = __builtin_amdgcn_readfirstlane((unsigned)(unsigned long long)lds);
    ab = uniform_ptr(ab); wb = uniform_ptr(wb);
    wid = __builtin_amdgcn_readfirstlane(wid); a_step = __builtin_amdgcn_readfirstlane(a_step);
    const int wm = wid / WN, wn = wid % WN;
    const int li = lane & 15, q = lane >> 4;
    auto wswz = [&](int k) { return (unsigned)(k * NFULL + (((lane & 7) ^ (((k >> 1) & 3) | (((k >> 4) & 1) << 2))) * 16)); };
    const unsigned b_voff0 = wswz((BIG ? wid * 8 : wid * 16) + (lane >> 3)), b_voff1 = wswz(wid * 16 + 8 + (lane >> 3));
    unsigned dA = lds0 + C::A_OFF + wid * (BIG ? 2048 : 1024), dB = lds0 + C::B_OFF + wid * (BIG ? 1024 : 2048);
    asm volatile("" : "+s"(dA), "+s"(dB));
    LAS char* ra0 = lds + C::A_OFF + (BIG ? (unsigned)((wm * MT * 16 + li) * 64 + ((q ^ ((-(li >> 2)) & 3)) * 16))
                                          : (unsigned)((wm * MT * 16 + li) * 128 + ((q ^ ((li >> 1) & 7)) * 16)));
    LAS char* ra1 = ra0 + (((q ^ ((li >> 1) & 7)) & 4) ? -64 : 64);
    LAS char* rb_[NT];
    const int fb = ((li >> 2) & 3) | ((q & 1) << 2);
#pragma unroll
    for (int n = 0; n < NT; ++n) rb_[n] = lds + C::B_OFF + (unsigned)((16 * q + (li >> 1)) * 128 + (((wn * NT + n) ^ fb) * 16) + 8 * (li & 1));
    v4i acc[MT][NT];
#pragma unroll
    for (int m = 0; m < MT; ++m)
#pragma unroll
        for (int n = 0; n < NT; ++n) acc[m][n] = (v4i){0, 0, 0, 0};

    auto dma = [&](auto ic, int sl) {
        constexpr int i = decltype(ic)::value;
        if constexpr (i == 0) lds_dma(a_voff0, ab, dA + (unsigned)(sl * A_BYTES));
        if constexpr (i == 1 && BIG) lds_dma(a_voff1, ab, dA + (unsigned)(sl * A_BYTES + 1024));
        if constexpr (i == 1 && !BIG) lds_dma(b_voff0, wb, dB + (unsigned)(sl * B_BYTES));
        if constexpr (i == 2 && BIG) lds_dma(b_voff0, wb, dB + (unsigned)(sl * B_BYTES));
        if constexpr (i == 2 && !BIG) lds_dma(b_voff1, wb, dB + (unsigned)(sl * B_BYTES + 1024));
        if constexpr (i == L - 1) { ab += a_step; wb += (BIG ? 64 : 128) * NFULL; }
    };
    auto issue = [&](int sl) { static_for<L>([&](auto ic) { dma(ic, sl); }); };
    auto step = [&](auto slotc, auto islotc, auto wvc) {
        constexpr int sl = decltype(slotc)::value, isl = decltype(islotc)::value, WV = decltype(wvc)::value;
        v4i wf[NSUB][NT], xf[NSUB][MT];
        auto rdw = [&](int s, int n) {
            const v2i lo = __builtin_amdgcn_ds_read_tr8_b64_v2i32((LAS v2i*)(rb_[n] + sl * B_BYTES + s * 8192));
            const v2i hi = __builtin_amdgcn_ds_read_tr8_b64_v2i32((LAS v2i*)(rb_[n] + sl * B_BYTES + s * 8192 + 1024));
            wf[s][n] = (v4i){lo.x, lo.y, hi.x, hi.y}; };
        auto rdx = [&](int s, int m) { xf[s][m] = *(LAS v4i*)((s ? ra1 : ra0) + sl * A_BYTES + m * (BIG ? 1024 : 2048)); };
        auto idma = [&](auto ic) { __builtin_amdgcn_sched_barrier(0); if constexpr (isl >= 0) dma(ic, isl); __builtin_amdgcn_sched_barrier(0); };
        if constexpr (BIG) {
            rdw(0, 0); rdw(0, 1); idma(IC<0>{});
            rdw(0, 2); rdw(0, 3); idma(IC<1>{});
            rdx(0, 0); rdx(0, 1); rdx(0, 2); rdx(0, 3); idma(IC<2>{});
        } else {
            rdw(0, 0); rdw(0, 1); idma(IC<0>{});
            rdx(0, 0); rdx(0, 1); rdw(1, 0); idma(IC<1>{});
            rdw(1, 1); rdx(1, 0); rdx(1, 1); idma(IC<2>{});
        }
        if constexpr (WV >= 0) waitv<WV * L>();
        asm volatile("s_waitcnt lgkmcnt(0)" ::: "memory");
        bar();
        __builtin_amdgcn_sched_barrier(0);
        __builtin_amdgcn_s_setprio(1);
#pragma unroll
        for (int s = 0; s < NSUB; ++s)
#pragma unroll
            for (int m = 0; m < MT; ++m)
#pragma unroll
                for (int n = 0; n < NT; ++n) acc[m][n] = __builtin_amdgcn_mfma_i32_16x16x64_i8(wf[s][n], xf[s][m], acc[m][n], 0, 0, 0);
        __builtin_amdgcn_s_setprio(0);
        __builtin_amdgcn_sched_barrier(0);
        bar();
    };
#pragma unroll
    for (int s = 0; s < PF; ++s) issue(s);
    waitv<(PF - 1) * L>();
    bar();
    if constexpr (SET == 1) bar();
    for (int j = 0; j < NLOOP; j += RING)
        static_for<RING>([&](auto ic) { constexpr int i = decltype(ic)::value; step(IC<i>{}, IC<(i + PF) % RING>{}, IC<PF - 1>{}); });
    static_for<C::NSTEP - NLOOP>([&](auto ic) {
        constexpr int j = NLOOP + decltype(ic)::value, st = C::stage_at(j);
        step(IC<j % RING>{}, IC<(st >= 0 ? st % RING : -1)>{}, IC<(j + 1 < C::NSTEP ? C::issued_upto(j) - (j + 1) : -1)>{}); });
    if constexpr (SET == 0) bar();
    const bool bad = __builtin_amdgcn_readfirstlane(((volatile LAS unsigned*)(lds + CTL_OFF))[1]) != 0u;
    float* op = out + (size_t)(wm * MT * 16 + (li & 7)) * NLOC + wn * NT * 16 + (li >> 3) * 16 + 4 * q;
    if (!bad) {
#pragma unroll
        for (int m = 0; m < MT; ++m)
#pragma unroll
            for (int n = 0; n < NT; n += 2) {
                v4i lo, hi;
#pragma unroll
                for (int e = 0; e < 4; ++e) {
                    lo[e] = __builtin_amdgcn_update_dpp(acc[m][n][e], acc[m][n + 1][e], 0x128, 0xf, 0xc, false);
                    hi[e] = __builtin_amdgcn_update_dpp(acc[m][n + 1][e], acc[m][n][e], 0x128, 0xf, 0x3, false);
                }
                v4f o; o.x = silu_dq(lo.x, scale); o.y = silu_dq(lo.y, scale); o.z = silu_dq(lo.z, scale); o.w = silu_dq(lo.w, scale);
                store_out(op + (size_t)m * 16 * NLOC + n * 16, o);
                o.x = silu_dq(hi.x, scale); o.y = silu_dq(hi.y, scale); o.z = silu_dq(hi.z, scale); o.w = silu_dq(hi.w, scale);
                store_out(op + (size_t)(m * 16 + 8) * NLOC + n * 16, o);
            }
    } else {
        const float qnan = __builtin_nanf("");
        for (int m = 0; m < MT; ++m)
            for (int n = 0; n < NT; n += 2) {
                store_out(op + (size_t)m * 16 * NLOC + n * 16, (v4f){qnan, qnan, qnan, qnan});
                store_out(op + (size_t)(m * 16 + 8) * NLOC + n * 16, (v4f){qnan, qnan, qnan, qnan});
            }
    }
}
template <bool BIG>
__device__ __forceinline__ void gemm_tile(LAS char* lds, int wid, int lane, const char* ab, unsigned a_voff0, unsigned a_voff1, int a_step,
                                          const char* wb, float* out, float scale) {
    wid = __builtin_amdgcn_readfirstlane(wid);
    if (wid < 4) gemm_tile_set<BIG, 0>(lds, wid, lane, ab, a_voff0, a_voff1, a_step, wb, out, scale);
    else gemm_tile_set<BIG, 1>(lds, wid, lane, ab, a_voff0, a_voff1, a_step, wb, out, scale);
}

__global__ __launch_bounds__(576) void fused_kernel(Params p) {
    extern __shared__ __attribute__((aligned(16))) char shm[];
    LAS char* lds = (LAS char*)shm;
    volatile LAS unsigned* ctl = (volatile LAS unsigned*)(lds + CTL_OFF);
    const int tid = threadIdx.x, lane = tid & 63, wid = __builtin_amdgcn_readfirstlane(tid >> 6);
    const int b = blockIdx.x, grp = b & 7, slot = b >> 3;
    char* myws = p.ws[p.rank];

    if (slot < 4) {
        if (wid >= 4) return;
        unsigned tkv; unsigned long long sv;
        asm volatile("s_mov_b64 %1, exec\n\ts_mov_b64 exec, 1\n\tglobal_atomic_add %0, %2, %3, %4 sc0\n\ts_mov_b64 exec, %1"
                     : "=&v"(tkv), "=&s"(sv) : "v"(0u), "v"(1u), "s"(myws + WS_TICKET + grp * 128) : "memory");
        if (grp == 7) { waitv<0>(); asm volatile("" :: "v"(tkv)); return; }
        ctl[16 + tid % 64] = 0u;
        asm volatile("s_waitcnt lgkmcnt(0)" ::: "memory");
        const int gw = slot * 4 + wid;
        const int dstr = (p.rank + 1 + grp) & 7;
        const char* xs = p.x;
        const int lr = lane >> 3, c8 = lane & 7;
        const unsigned soff = (unsigned)((16 * gw + lr) * KTOT + c8 * 16);
        unsigned doff[2];
#pragma unroll
        for (int i = 0; i < 2; ++i) { const int r = 16 * gw + 8 * i + lr;
            doff[i] = (unsigned)((c8 >> 2) * IMG_BYTES + r * 64 + (((c8 & 3) ^ ((-(r >> 2)) & 3)) * 16)); }
        v4u R[PW][2];
        auto ld = [&](v4u (&r)[2], int c) {
            const int cc = c < NCHUNK ? c : NCHUNK - 1;
            const char* sp = xs + (unsigned)(order_rb(cc) * 256 * KTOT + order_kt(cc) * 128);
            asm volatile("global_load_dwordx4 %0, %1, %2" : "=v"(r[0]) : "v"(soff), "s"(sp) : "memory");
            asm volatile("global_load_dwordx4 %0, %1, %2" : "=v"(r[1]) : "v"(soff + 8u * KTOT), "s"(sp) : "memory"); };
#pragma unroll
        for (int i = 0; i < PW; ++i) ld(R[i], i);
        unsigned tk;
        asm volatile("s_waitcnt vmcnt(0)\n\tv_readfirstlane_b32 %0, %1" : "=s"(tk) : "v"(tkv) : "memory");
        bar();
        const unsigned call = tk / TICKETS + 1u;
        char* dst = p.ws[dstr] + WS_X + ((size_t)(call & 1u) * NR + p.rank) * SHARD_BYTES;
        char* flag = p.ws[dstr] + WS_FLAG + (size_t)p.rank * NCHUNK * 128 + slot * 4;
        const unsigned cnt0 = (unsigned)(unsigned long long)lds + CNT_OFF;
        auto publish = [&](int t) {
            unsigned old = 0;
            if (t >= 0) {
                if (lane == 0) asm volatile("ds_add_rtn_u32 %0, %1, %2\n\ts_waitcnt lgkmcnt(0)" : "=&v"(old) : "v"(cnt0 + (unsigned)t * 4u), "v"(1u) : "memory");
                old = __builtin_amdgcn_readfirstlane(old);
            }
            char* fp = (t >= 0 && old == 3u) ? flag + (size_t)t * 128 : myws + WS_DUMMY + (b * 4 + wid) * 128;
            if (lane == 0) asm volatile("global_store_dword %0, %1, off sc0 sc1" :: "v"(fp), "v"(call) : "memory"); };
        static_assert(NCHUNK % PW == 0, "unroll");
        for (int c0 = 0; c0 < NCHUNK; c0 += PW) static_for<PW>([&](auto ic) {
            constexpr int i = decltype(ic)::value;
            const int c = c0 + i;
            waitv<5 * (PW - 1)>();
            publish(c - PW);
            const char* dp = dst + (unsigned)(order_rb(c) * RB_BYTES + order_kt(c) * 2 * IMG_BYTES);
            asm volatile("" : "+v"(R[i][0]), "+v"(R[i][1]));
            asm volatile("global_store_dwordx4 %0, %1, %2 sc0 sc1\n\ts_nop 1" :: "v"(doff[0]), "v"(R[i][0]), "s"(dp) : "memory");
            asm volatile("global_store_dwordx4 %0, %1, %2 sc0 sc1\n\ts_nop 1" :: "v"(doff[1]), "v"(R[i][1]), "s"(dp) : "memory");
            ld(R[i], c + PW);
        });
        static_for<PW>([&](auto ic) {
            constexpr int v = decltype(ic)::value;
            waitv<2 + 5 * (PW - 1 - v) + v>();
            publish(NCHUNK - PW + v);
        });
        waitv<0>();
#pragma unroll
        for (int i = 0; i < PW; ++i) asm volatile("" :: "v"(R[i][0]), "v"(R[i][1]));
        if (wid == 0) STAMP(7);
        return;
    }

    const int j = slot - 4, pj = j >> 2;
    const int h = grp >> 2;
    const int rb = (h ^ pj) & 1;
    const int src = (p.rank + 1 + pj) & 7;
    const int cb = (grp & 3) * 4 + (j & 3);
    const bool has_own = pj < 4;

    if (wid == 8) {
        if (slot == 31) { __builtin_amdgcn_fence(__ATOMIC_ACQUIRE, ""); waitv<0>(); }
        unsigned tk = 0;
        if (lane == 0) tk = __hip_atomic_fetch_add((unsigned*)(myws + WS_TICKET + grp * 128), 1u, __ATOMIC_RELAXED, __HIP_MEMORY_SCOPE_AGENT);
        tk = __builtin_amdgcn_readfirstlane(tk);
        const unsigned call = tk / TICKETS + 1u;
        const char* flag = myws + WS_FLAG + (size_t)src * NCHUNK * 128 + (lane & 3) * 4;
        unsigned* tmo = (unsigned*)(myws + WS_TMO);
        int ready = 0;
        bool bad = false;
        auto poll = [&]() -> unsigned {
            int kt = ready + (lane >> 2); kt = kt < 32 ? kt : 31;
            const unsigned* q = lane < 60 ? (const unsigned*)(flag + order_pos(rb, kt) * 128) : tmo;
            const unsigned v = __hip_atomic_load(q, __ATOMIC_RELAXED, __HIP_MEMORY_SCOPE_SYSTEM);
            unsigned long long m = __ballot(v >= call);
            m = m & (m >> 1); m = m & (m >> 2);
            m = (~m & 0x1111111111111111ull) | (1ull << 60);
            ready += (int)(__builtin_ctzll(m) >> 2);
            ready = ready < 32 ? ready : 32;
            return __builtin_amdgcn_readlane(v, 60);
        };
        auto ensure = [&](int kt) {
            const int need = kt + 1;
            if (ready >= need) return;
            const unsigned long long t0 = __builtin_amdgcn_s_memrealtime();
            for (;;) {
                const unsigned tv = poll();
                if (ready >= need) {
                    break; }
                if (tv == call || __builtin_amdgcn_s_memrealtime() - t0 > DEADLINE) {
                    if (lane == 0) __hip_atomic_store(tmo, call, __ATOMIC_RELAXED, __HIP_MEMORY_SCOPE_AGENT);
                    bad = true; ready = 32; break;
                }
            }
        };
        if (lane == 0) { ctl[0] = call; ctl[1] = 0u; }
        asm volatile("s_waitcnt lgkmcnt(0)" ::: "memory");
        unsigned wo;
        asm volatile("global_load_dword %0, %1, off" : "=v"(wo) : "v"(p.out + (size_t)(src * MLOC + rb * 256 + lane * 4) * NLOC + cb * 128) : "memory");
        if (has_own) for (int i = 0; i < Cfg<false>::NBAR; ++i) bar();
        waitv<0>();
        asm volatile("" :: "v"(wo));
        using CB = Cfg<true>;
        if (has_own) poll();
        ensure(CB::PF >> 1);
        bar(); bar();
        for (int s = 0; s < CB::NSTEP; ++s) {
            bar();
            const int st = CB::stage_at(s + 1);
            if (st >= 0) ensure(st >> 1);
            if (s == CB::NSTEP - 1) { if (lane == 0) ctl[1] = bad ? 1u : 0u; asm volatile("s_waitcnt lgkmcnt(0)" ::: "memory"); }
            bar();
        }
        bar();
        STAMP(6);
        return;
    }

    const float scale = (p.sx[0] * p.sw[0]) * -1.44269504088896341f;
    const char* wb = p.w + (size_t)p.rank * NLOC + cb * 128;
    if (has_own) {
        const int row0 = (h * 4 + pj) * 64, r = wid * 8 + (lane >> 3);
        gemm_tile<false>(lds, wid, lane, p.x + (size_t)row0 * KTOT, (unsigned)(r * KTOT + (((lane & 7) ^ ((r >> 1) & 7)) * 16)), 0u, 128,
                         wb, p.out + (size_t)(p.rank * MLOC + row0) * NLOC + cb * 128, scale);
        if (wid == 0) STAMP(0);
    }
    unsigned wr[5] = {0u, 0u, 0u, 0u, 0u};
    if (!has_own) {
        __builtin_amdgcn_s_sleep(16);
        const int ww = (j - 16) * 8 + wid;
        const char* wq = p.w + (size_t)p.rank * NLOC + (grp & 3) * 512 + (size_t)(lane >> 2) * NFULL + (lane & 3) * 128;
        const char* xq = p.x + (size_t)h * 256 * KTOT + lane * 128;
#pragma unroll
        for (int i = 0; i < 3; ++i) {
            const int t = ww + 96 * i;
            asm volatile("global_load_dword %0, %1, off" : "=v"(wr[i]) : "v"(wq + (size_t)(t < 256 ? t : 255) * 16 * NFULL) : "memory");
            if (i < 2) asm volatile("global_load_dword %0, %1, off" : "=v"(wr[3 + i]) : "v"(xq + (size_t)(t < 128 ? t : 127) * 8192) : "memory");
        }
    }
    bar();
    waitv<0>();
#pragma unroll
    for (int i = 0; i < 5; ++i) asm volatile("" :: "v"(wr[i]));
    const char* xsrc = p.ws[p.rank] + WS_X + ((size_t)(ctl[0] & 1u) * NR + src) * SHARD_BYTES + (size_t)rb * RB_BYTES;
    const unsigned lin0 = (unsigned)(wid * 2048 + lane * 16);
    gemm_tile<true>(lds, wid, lane, xsrc, lin0, lin0 + 1024, IMG_BYTES, wb, p.out + (size_t)(src * MLOC + rb * 256) * NLOC + cb * 128, scale);
}

}

extern "C" void kernel_launch(int rank, int nranks, ncclComm_t comm,
                              hipStream_t stream,
                              void* const* d_in, const int* in_sizes, int n_in,
                              void* d_out, int out_size,
                              const int* in_dtypes, int out_dtype,
                              void* const* d_out_all,
                              void* const* d_ws_all, size_t ws_bytes) {
    (void)comm; (void)in_sizes; (void)n_in; (void)out_size; (void)in_dtypes; (void)out_dtype; (void)d_out_all;
    if (nranks != NR || rank < 0 || rank >= NR || ws_bytes < WS_NEED) { fprintf(stderr, "unexpected configuration\n"); return; }
    HIP_CHECK(hipFuncSetAttribute((const void*)fused_kernel, hipFuncAttributeMaxDynamicSharedMemorySize, LDS_BYTES));
    Params p;
    p.x = (const char*)d_in[0]; p.w = (const char*)d_in[1]; p.sx = (const float*)d_in[2]; p.sw = (const float*)d_in[3];
    p.out = (float*)d_out; p.rank = rank;
    for (int i = 0; i < NR; ++i) p.ws[i] = (char*)d_ws_all[i];
    hipLaunchKernelGGL(fused_kernel, dim3(256), dim3(576), LDS_BYTES, stream, p);
    HIP_CHECK(hipGetLastError());
}
```
